```python
import numpy as np

kernel_name = "gpu_ag_gemm_m4096_k4096_n4096_e4m3e5m2_dqrelu"


ABSMAX_RTOL = 0.05

COMPUTE_DTYPE = "float32"
EPILOGUE = "dequant_relu"
IN_QUANT = None
OUT_QUANT = None
IN_FP8 = "e4m3e5m2"
IN_DTYPES = {"x": "e4m3", "w_mat": "e5m2",
             "scale_x": "float32", "scale_w": "float32"}
WORLD_SIZE = 8

def _snap_fp8(x, mant_bits, max_abs, min_normal):
    a = np.abs(x.astype(np.float64))
    _m, _e = np.frexp(a)
    q = np.where(a >= min_normal, np.ldexp(1.0, _e - 1 - mant_bits),
                 min_normal * 2.0 ** -mant_bits)
    snapped = np.minimum(np.round(a / q) * q, max_abs)
    return (np.sign(x) * snapped).astype(np.float32)

def setup_inputs():

    x = _snap_fp8(np.random.randn(4096, 4096).astype(np.float32), 3, 448.0, 2.0 ** -6)
    w_mat = _snap_fp8(np.random.randn(4096, 4096).astype(np.float32), 2, 57344.0, 2.0 ** -14)
    scale_x = (0.02 * (1.0 + np.random.rand(1))).astype(np.float32)
    scale_w = (0.02 * (1.0 + np.random.rand(1))).astype(np.float32)
    return {"x": x, "w_mat": w_mat, "scale_x": scale_x, "scale_w": scale_w}

def reference(x, w_mat, scale_x, scale_w):
    acc = (x @ w_mat).astype(np.float32)
    y = (acc * (scale_x * scale_w)).astype(np.float32)
    return np.maximum(y, 0.0).astype(np.float32)
```

```cpp
#include <hip/hip_runtime.h>
#include <rccl/rccl.h>
#include <cstdio>
#include <cstdint>

#define HIP_CHECK(expr) do { hipError_t _e = (expr); if (_e != hipSuccess) { \
    fprintf(stderr, "HIP error %s at %s:%d\n", hipGetErrorString(_e), __FILE__, __LINE__); \
    return; } } while (0)

namespace {

typedef int v2i __attribute__((ext_vector_type(2)));
typedef int v4i __attribute__((ext_vector_type(4)));
typedef int v8i __attribute__((ext_vector_type(8)));
typedef unsigned v4u __attribute__((ext_vector_type(4)));
typedef float v4f __attribute__((ext_vector_type(4)));
#define LAS __attribute__((address_space(3)))

constexpr int NR = 8;
constexpr int KD = 4096, NL = 512, MS = 512;
constexpr int TM = 64, TN = 128, KC = 256;
constexpr int NKC = KD / KC;
constexpr int NRB = MS / TM;
constexpr int NWG = 256;
constexpr int STAGES = 3;
constexpr int XT = TM * KC;
constexpr int WT = KC * TN;
constexpr int LDS_X = 0;
constexpr int LDS_W = STAGES * XT;
constexpr int LDS_CTL = STAGES * (XT + WT);
constexpr int LDS_BYTES = LDS_CTL + 256;
constexpr int C_FULLX = 0, C_FULLW = 4, C_FREE = 8  , C_BAD = 12;

#define PDEPTH 2
#define INFL 2
#define PWAVES 1
#define NPWX 4
constexpr int NPWG = 8 * NPWX;
#define TAILK 1
#define PROWS_ 8
constexpr int PROWS = PROWS_;
constexpr int PLD = PROWS / 4;
constexpr int PPU = TM / PROWS;
constexpr int NPIECE = NRB * NKC * PPU;

constexpr size_t WS_CTR_L = 0;
constexpr size_t WS_CTR_P = 4096;
constexpr size_t WS_TMO = 24576;
constexpr size_t WS_TICK = 28672;
constexpr size_t WS_FLAGS = 65536;
constexpr size_t WS_DBG = 512u << 10;
constexpr size_t WS_XBUF = 1u << 20;
constexpr size_t XBUF_BYTES = (size_t)NR * MS * KD;
constexpr size_t WS_NEED = WS_XBUF + 2 * XBUF_BYTES;
constexpr unsigned long long DEADLINE_TICKS = 40000000ull;

struct Params {
  const unsigned char* x;
  const unsigned char* w;
  const float* sx;
  const float* sw;
  float* out;
  unsigned char* ws[NR];
  int rank;
};

__device__ __forceinline__ void glds16(const void* gsrc, unsigned lds_dst) {
  unsigned keep;
  asm volatile("s_mov_b32 %0, m0\n\ts_mov_b32 m0, %2\n\ts_nop 0\n\tglobal_load_lds_dwordx4 %1, off\n\ts_mov_b32 m0, %0"
               : "=&s"(keep) : "v"(gsrc), "s"(lds_dst) : "memory");
}
__device__ __forceinline__ void glds16_sc1(const void* gsrc, unsigned lds_dst) {
  unsigned keep;
  asm volatile("s_mov_b32 %0, m0\n\ts_mov_b32 m0, %2\n\ts_nop 0\n\tglobal_load_lds_dwordx4 %1, off sc1\n\ts_mov_b32 m0, %0"
               : "=&s"(keep) : "v"(gsrc), "s"(lds_dst) : "memory");
}
__device__ __forceinline__ unsigned lds_rd(unsigned addr) {
  unsigned v;
  asm volatile("ds_read_b32 %0, %1\n\ts_waitcnt lgkmcnt(0)" : "=v"(v) : "v"(addr) : "memory");
  return __builtin_amdgcn_readfirstlane(v);
}
__device__ __forceinline__ void lds_wr(unsigned addr, unsigned v) {
  asm volatile("ds_write_b32 %0, %1\n\ts_waitcnt lgkmcnt(0)" :: "v"(addr), "v"(v) : "memory");
}
__device__ __forceinline__ void st16_sys(void* p, v4u v) {
  asm volatile("global_store_dwordx4 %0, %1, off sc0 sc1\n\ts_nop 1" :: "v"(p), "v"(v) : "memory");
}
__device__ __forceinline__ void st4_sys(void* p, unsigned v) {
  asm volatile("global_store_dword %0, %1, off sc0 sc1" :: "v"(p), "v"(v) : "memory");
}
#define WAIT_VM(n) asm volatile("s_waitcnt vmcnt(" #n ")" ::: "memory")
template <int N> __device__ __forceinline__ void wait_vm() {
  asm volatile("s_waitcnt vmcnt(%0)" :: "i"(N > 63 ? 63 : N) : "memory");
}
template <int V> struct IC { static constexpr int value = V; };
template <int A, int B, class F> __device__ __forceinline__ void static_for(F&& f) {
  if constexpr (A < B) { f(IC<A>{}); static_for<A + 1, B>(f); }
}
constexpr int FLAGOPS = 1;

#define STAMP(slot) do {} while (0)
__device__ __forceinline__ void unit_of(int u, int& rb, int& kc) {
  constexpr int HEAD = (NKC - TAILK) * NRB;
  if (u < HEAD) { kc = u / NRB; rb = u % NRB; }
  else { int v = u - HEAD; rb = v / TAILK; kc = (NKC - TAILK) + v % TAILK; }
}

__global__ void __launch_bounds__(448 * PWAVES, 2 * PWAVES) ag_gemm_kernel(Params p) {
  extern __shared__ __attribute__((aligned(16))) unsigned char smem[];
  const int tid = threadIdx.x;
  const int wid = __builtin_amdgcn_readfirstlane(tid >> 6);
  const int lane = tid & 63;
  const int bid = blockIdx.x;
  const int rank = p.rank;
  const unsigned lds0 = (unsigned)(uintptr_t)(LAS unsigned char*)smem;
  const unsigned ctl = lds0 + LDS_CTL;
  unsigned char* myws = p.ws[rank];

  const int xcd = bid & 7;
  const int idx = bid >> 3;
  STAMP(0);

  if (idx >= 28) {
    if (idx < 32 - NPWX) return;
    const int lr = lane >> 4, lc = lane & 15;
    const int pj = wid % 7;
    unsigned char* peer = p.ws[(rank + 1 + pj) & 7];
    unsigned* ctr = (unsigned*)(myws + WS_CTR_P) + bid * 16 + wid;
    unsigned* tick = (unsigned*)(myws + WS_TICK + pj * 128);
    v4u d[2][PLD];
    auto load = [&](int pc, v4u* dd) {
      int rbb, kcc; unit_of(pc / PPU, rbb, kcc);
      const unsigned char* s = p.x + (size_t)(rbb * TM + (pc % PPU) * PROWS + lr) * KD + kcc * KC + lc * 16;
      #pragma unroll
      for (int u = 0; u < PLD; ++u)
        asm volatile("global_load_dwordx4 %0, %1, off" : "=v"(dd[u]) : "v"(s + (size_t)u * 4 * KD) : "memory");
    };
    auto grab = [&](unsigned& t) {
      unsigned one = 1;
      if (lane == 0) asm volatile("global_atomic_add %0, %1, %2, off sc0" : "=v"(t) : "v"(tick), "v"(one) : "memory");
    };
    unsigned c, ta = 0, tb = 0;
    asm volatile("global_load_dword %0, %1, off sc1" : "=v"(c) : "v"(ctr) : "memory");
    grab(ta); grab(tb);
    wait_vm<0>();
    asm volatile("" : "+v"(c), "+v"(ta), "+v"(tb));
    const unsigned call = __builtin_amdgcn_readfirstlane(c) + 1;
    const unsigned base = (call - 1) * (unsigned)(NPIECE + 2 * NPWG * PWAVES);
    const size_t bufoff = WS_XBUF + (size_t)(call & 1) * XBUF_BYTES;
    unsigned cur = __builtin_amdgcn_readfirstlane(ta) - base;
    unsigned nxt = __builtin_amdgcn_readfirstlane(tb) - base;
    STAMP(1);
    if (cur < (unsigned)NPIECE) { load(cur, d[0]); wait_vm<0>(); }
    auto step = [&](v4u* dc, v4u* dn) {
      #pragma unroll
      for (int u = 0; u < PLD; ++u) asm volatile("" : "+v"(dc[u]));
      unsigned t = 0;
      const int pc = cur;
      int rbb, kcc; unit_of(pc / PPU, rbb, kcc);
      unsigned char* dst = peer + bufoff + (size_t)((rank * NRB + rbb) * NKC + kcc) * XT + (pc % PPU) * (PROWS * KC) + lane * 16;
      constexpr int F0 = INFL < PLD ? INFL : PLD;
      #pragma unroll
      for (int u = 0; u < F0; ++u) st16_sys(dst + u * 1024, dc[u]);
      grab(t);
      if (nxt < (unsigned)NPIECE) load(nxt, dn);
      static_for<F0, PLD>([&](auto uc) {
        constexpr int u = decltype(uc)::value;
        wait_vm<(u - INFL < F0) ? INFL - 1 + 1 + PLD : INFL - 1>();
        st16_sys(dst + u * 1024, dc[u]);
      });
      wait_vm<0>();
      asm volatile("" : "+v"(t));
      if (lane == 0) st4_sys(peer + WS_FLAGS + ((size_t)((rank * NRB + rbb) * NKC + kcc) * PPU + (pc % PPU)) * 4, call);
      cur = nxt;
      nxt = __builtin_amdgcn_readfirstlane(t) - base;
    };
    while (cur < (unsigned)NPIECE) {
      step(d[0], d[1]);
      if (cur >= (unsigned)NPIECE) break;
      step(d[1], d[0]);
    }
    STAMP(2);
    if (lane == 0) asm volatile("global_store_dword %0, %1, off sc1" :: "v"(ctr), "v"(call) : "memory");
    return;
  }
  if (wid >= 6) return;

  const int nt = idx & 3;
  const int jj = idx >> 2;
  const int R = xcd * 7 + jj;
  const int rsrc = (rank + 1 + (R >> 3)) & 7;
  const int rrb = R & 7;
  const int npass = (jj == 0) ? 2 : 1;

  if (tid < 16) *(LAS unsigned*)(smem + LDS_CTL + tid * 4) = 0;
  asm volatile("s_waitcnt lgkmcnt(0)" ::: "memory");
  __builtin_amdgcn_s_barrier();

  const int ntot = npass * NKC;
  auto wait_free = [&](int c) {
    if (c >= STAGES) {
      for (;;) {
        unsigned f0 = lds_rd(ctl + (C_FREE + 0) * 4), f1 = lds_rd(ctl + (C_FREE + 1) * 4);
        unsigned f2 = lds_rd(ctl + (C_FREE + 2) * 4), f3 = lds_rd(ctl + (C_FREE + 3) * 4);
        unsigned m = min(min(f0, f1), min(f2, f3));
        if (m >= (unsigned)(c - STAGES + 1)) break;
        __builtin_amdgcn_s_sleep(1);
      }
    }
  };

  if (wid == 4) {
    unsigned* ctr = (unsigned*)(myws + WS_CTR_L) + bid;
    unsigned call;
    {
      unsigned c;
      asm volatile("global_load_dword %0, %1, off sc1\n\ts_waitcnt vmcnt(0)" : "=v"(c) : "v"(ctr) : "memory");
      call = __builtin_amdgcn_readfirstlane(c) + 1;
      if (lane == 0) asm volatile("global_store_dword %0, %1, off sc1" :: "v"(ctr), "v"(call) : "memory");
    }
    const unsigned char* fl = myws + WS_FLAGS + (size_t)(rsrc * NRB + rrb) * NKC * PPU * 4;
    unsigned* tmo = (unsigned*)(myws + WS_TMO);
    const int lr = lane >> 4, lc = lane & 15;
    bool dead = false;
    const unsigned long long t0 = __builtin_amdgcn_s_memrealtime();
    for (int c = 0; c < ntot; ++c) {
      const int slot = c % STAGES;
      const bool own = (npass == 2 && c < NKC);
      const int kc = c & (NKC - 1);
      const unsigned char* xbase = own ? p.x + (size_t)xcd * TM * KD + kc * KC
                                       : myws + WS_XBUF + (size_t)(call & 1) * XBUF_BYTES + (size_t)((rsrc * NRB + rrb) * NKC + kc) * XT;
      const int rstride = own ? KD : KC;
      wait_free(c);
      if (!own) {
        bool first = true;
        for (;;) {
          unsigned f, tv;
          asm volatile("global_load_dword %0, %2, off sc0 sc1\n\tglobal_load_dword %1, %3, off sc1\n\ts_waitcnt vmcnt(0)"
                       : "=&v"(f), "=&v"(tv) : "v"(fl + (kc * PPU + (lane & (PPU - 1))) * 4), "v"(tmo) : "memory");
          if (first) { first = false; if (c > 0) lds_wr(ctl + (C_FULLX + (c - 1) % STAGES) * 4, c); }
          const unsigned m = __all(f >= call) ? call : 0;
          if (m >= call || dead) break;
          if (__builtin_amdgcn_readfirstlane(tv) == call || __builtin_amdgcn_s_memrealtime() - t0 > DEADLINE_TICKS) {
            if (lane == 0) asm volatile("global_store_dword %0, %1, off sc1" :: "v"(tmo), "v"(call) : "memory");
            lds_wr(ctl + C_BAD * 4, 1);
            dead = true;
            break;
          }
          __builtin_amdgcn_s_sleep(2);
        }
      }
      if (c == ntot - 1) STAMP(1);
      if (c == ntot - TAILK) STAMP(3);
      if (c == ntot - NKC) STAMP(4);
      const unsigned dst = lds0 + LDS_X + slot * XT;
      #pragma unroll
      for (int i = 0; i < 16; ++i) {
        const int row = 4 * i + lr;
        const unsigned char* g = xbase + row * rstride + ((lc ^ (row & 15)) * 16);
        glds16_sc1(g, dst + i * 1024);
      }
      if (own && c > 0) {
        WAIT_VM(16); lds_wr(ctl + (C_FULLX + (c - 1) % STAGES) * 4, c);
      }
    }
    WAIT_VM(0);
    lds_wr(ctl + (C_FULLX + (ntot - 1) % STAGES) * 4, ntot);
    STAMP(2);
    return;
  }

  if (wid == 5) {
    const unsigned char* wbase = p.w + nt * TN;
    const int lr = lane >> 3, lc = lane & 7;
    for (int c = 0; c < ntot; ++c) {
      const int slot = c % STAGES;
      const int kc = c & (NKC - 1);
      wait_free(c);
      const unsigned dst = lds0 + LDS_W + slot * WT;
      #pragma unroll
      for (int i = 0; i < 32; ++i) {
        const int row = 8 * i + lr;
        const int h = ((row >> 1) & 3) | (((row >> 5) & 1) << 2);
        const unsigned char* g = wbase + (size_t)(kc * KC + row) * NL + ((lc ^ h) * 16);
        glds16(g, dst + i * 1024);
      }
      if (c > 0) { WAIT_VM(32); lds_wr(ctl + (C_FULLW + (c - 1) % STAGES) * 4, c); }
    }
    WAIT_VM(0);
    lds_wr(ctl + (C_FULLW + (ntot - 1) % STAGES) * 4, ntot);
    return;
  }

  const float scale = p.sx[0] * p.sw[0];
  const int g = lane >> 4, li = lane & 15;
  const int q = li >> 1, pp = li & 1;
  for (int pass = 0; pass < npass; ++pass) {
    v4f acc[4][2];
    #pragma unroll
    for (int m = 0; m < 4; ++m) { acc[m][0] = v4f{0, 0, 0, 0}; acc[m][1] = v4f{0, 0, 0, 0}; }
    for (int kc = 0; kc < NKC; ++kc) {
      const int c = pass * NKC + kc;
      const int slot = c % STAGES;
      for (;;) {
        unsigned fx = lds_rd(ctl + (C_FULLX + slot) * 4), fw = lds_rd(ctl + (C_FULLW + slot) * 4);
        if (min(fx, fw) >= (unsigned)(c + 1)) break;
        __builtin_amdgcn_s_sleep(1);
      }
      const LAS unsigned char* xs = (const LAS unsigned char*)smem + LDS_X + slot * XT;
      const LAS unsigned char* wsm = (const LAS unsigned char*)smem + LDS_W + slot * WT;
      #pragma unroll
      for (int ks = 0; ks < 2; ++ks) {
        v8i af[2], bf[4];
        #pragma unroll
        for (int n = 0; n < 2; ++n) {
          #pragma unroll
          for (int t = 0; t < 4; ++t) {
            const int k = 128 * ks + 32 * g + 8 * t + q;
            const int h = ((k >> 1) & 3) | (((k >> 5) & 1) << 2);
            v2i r = __builtin_amdgcn_ds_read_tr8_b64_v2i32((LAS v2i*)(wsm + k * 128 + (((2 * wid + n) ^ h) * 16) + 8 * pp));
            af[n][2 * t] = r[0]; af[n][2 * t + 1] = r[1];
          }
        }
        #pragma unroll
        for (int m = 0; m < 4; ++m) {
          const int row = 16 * m + li;
          v4i lo = *(const LAS v4i*)(xs + row * 256 + (((8 * ks + 2 * g) ^ li) * 16));
          v4i hi = *(const LAS v4i*)(xs + row * 256 + (((8 * ks + 2 * g + 1) ^ li) * 16));
          bf[m] = v8i{lo[0], lo[1], lo[2], lo[3], hi[0], hi[1], hi[2], hi[3]};
        }
        if (ks == 1) {
          asm volatile("s_waitcnt lgkmcnt(0)" ::: "memory");
          __builtin_amdgcn_sched_barrier(0);
          if (c + STAGES < ntot) lds_wr(ctl + (C_FREE + wid) * 4, c + 1);
        }
        #pragma unroll
        for (int m = 0; m < 4; ++m)
          #pragma unroll
          for (int n = 0; n < 2; ++n)
            acc[m][n] = __builtin_amdgcn_mfma_scale_f32_16x16x128_f8f6f4(af[n], bf[m], acc[m][n], 1, 0, 0, 0, 0, 0);
      }
    }
    STAMP(1);
    const bool own = (npass == 2 && pass == 0);
    const bool bad = !own && lds_rd(ctl + C_BAD * 4) != 0;
    const int orow = own ? rank * MS + xcd * TM : rsrc * MS + rrb * TM;
    float* obase = p.out + (size_t)(orow + li) * NL + nt * TN + 32 * wid + 4 * g;
    #pragma unroll
    for (int m = 0; m < 4; ++m)
      #pragma unroll
      for (int n = 0; n < 2; ++n) {
        v4f v = acc[m][n] * scale;
        v[0] = fmaxf(v[0], 0.f); v[1] = fmaxf(v[1], 0.f); v[2] = fmaxf(v[2], 0.f); v[3] = fmaxf(v[3], 0.f);
        if (bad) v = v4f{__builtin_nanf(""), __builtin_nanf(""), __builtin_nanf(""), __builtin_nanf("")};
        *(v4f*)(obase + (size_t)m * 16 * NL + n * 16) = v;
      }
    STAMP(2);
  }
}

}

extern "C" void kernel_launch(int rank, int nranks, ncclComm_t comm,
                              hipStream_t stream,
                              void* const* d_in, const int* in_sizes, int n_in,
                              void* d_out, int out_size,
                              const int* in_dtypes, int out_dtype,
                              void* const* d_out_all,
                              void* const* d_ws_all, size_t ws_bytes) {
  (void)comm; (void)in_sizes; (void)in_dtypes; (void)out_dtype; (void)d_out_all;
  if (nranks != NR || n_in != 4 || out_size != NR * MS * NL || ws_bytes < WS_NEED || (unsigned)rank >= (unsigned)NR) {
    fprintf(stderr, "kernel_launch: unexpected problem shape\n");
    return;
  }
  HIP_CHECK(hipFuncSetAttribute((const void*)ag_gemm_kernel, hipFuncAttributeMaxDynamicSharedMemorySize, LDS_BYTES));
  Params p;
  p.x = (const unsigned char*)d_in[0];
  p.w = (const unsigned char*)d_in[1];
  p.sx = (const float*)d_in[2];
  p.sw = (const float*)d_in[3];
  p.out = (float*)d_out;
  for (int i = 0; i < NR; ++i) p.ws[i] = (unsigned char*)d_ws_all[i];
  p.rank = rank;
  ag_gemm_kernel<<<NWG, 448 * PWAVES, LDS_BYTES, stream>>>(p);
  HIP_CHECK(hipGetLastError());
}
```

```cpp
#include <hip/hip_runtime.h>
#include <rccl/rccl.h>
#include <cstdio>
#include <cstdint>

#define HIP_CHECK(expr) do { hipError_t _e = (expr); if (_e != hipSuccess) { \
    fprintf(stderr, "HIP error %s at %s:%d\n", hipGetErrorString(_e), __FILE__, __LINE__); \
    return; } } while (0)

namespace {

typedef int v2i __attribute__((ext_vector_type(2)));
typedef int v4i __attribute__((ext_vector_type(4)));
typedef int v8i __attribute__((ext_vector_type(8)));
typedef unsigned v4u __attribute__((ext_vector_type(4)));
typedef float v4f __attribute__((ext_vector_type(4)));
#define LAS __attribute__((address_space(3)))

constexpr int NR = 8;
constexpr int KD = 4096, NL = 512, MS = 512;
constexpr int TM = 64, TN = 128, KC = 256;
constexpr int NKC = KD / KC;
constexpr int NRB = MS / TM;
constexpr int NWG = 256;
constexpr int STAGES = 3;
constexpr int XT = TM * KC;
constexpr int WT = KC * TN;
constexpr int LDS_X = 0;
constexpr int LDS_W = STAGES * XT;
constexpr int LDS_CTL = STAGES * (XT + WT);
constexpr int LDS_BYTES = LDS_CTL + 256;
constexpr int C_FULLX = 0, C_FULLW = 4, C_FREE = 8  , C_BAD = 12;

constexpr int NPWG = 32;
#define TAILK 1
#define GSZ 2
#define FINEK 3
constexpr int PPU = 8;
constexpr int NHEAD = (NKC - FINEK) * NRB * (PPU / GSZ);
constexpr int NGRP = NHEAD + FINEK * NRB * PPU;

constexpr size_t WS_CTR_L = 0;
constexpr size_t WS_CTR_P = 4096;
constexpr size_t WS_TMO = 24576;
constexpr size_t WS_TICK = 28672;
constexpr size_t WS_FLAGS = 65536;
constexpr size_t WS_DBG = 512u << 10;
constexpr size_t WS_XBUF = 1u << 20;
constexpr size_t XBUF_BYTES = (size_t)NR * MS * KD;
constexpr size_t WS_NEED = WS_XBUF + 2 * XBUF_BYTES;
constexpr unsigned long long DEADLINE_TICKS = 40000000ull;

struct Params {
  const unsigned char* x;
  const unsigned char* w;
  const float* sx;
  const float* sw;
  float* out;
  unsigned char* ws[NR];
  int rank;
};

__device__ __forceinline__ void glds16(const void* gsrc, unsigned lds_dst) {
  unsigned keep;
  asm volatile("s_mov_b32 %0, m0\n\ts_mov_b32 m0, %2\n\ts_nop 0\n\tglobal_load_lds_dwordx4 %1, off\n\ts_mov_b32 m0, %0"
               : "=&s"(keep) : "v"(gsrc), "s"(lds_dst) : "memory");
}
__device__ __forceinline__ void glds16_sc1(const void* gsrc, unsigned lds_dst) {
  unsigned keep;
  asm volatile("s_mov_b32 %0, m0\n\ts_mov_b32 m0, %2\n\ts_nop 0\n\tglobal_load_lds_dwordx4 %1, off sc1\n\ts_mov_b32 m0, %0"
               : "=&s"(keep) : "v"(gsrc), "s"(lds_dst) : "memory");
}
__device__ __forceinline__ unsigned lds_rd(unsigned addr) {
  unsigned v;
  asm volatile("ds_read_b32 %0, %1\n\ts_waitcnt lgkmcnt(0)" : "=v"(v) : "v"(addr) : "memory");
  return __builtin_amdgcn_readfirstlane(v);
}
__device__ __forceinline__ void lds_wr(unsigned addr, unsigned v) {
  asm volatile("ds_write_b32 %0, %1\n\ts_waitcnt lgkmcnt(0)" :: "v"(addr), "v"(v) : "memory");
}
__device__ __forceinline__ void st16_sys(void* p, v4u v) {
  asm volatile("global_store_dwordx4 %0, %1, off sc0 sc1\n\ts_nop 1" :: "v"(p), "v"(v) : "memory");
}
__device__ __forceinline__ void st4_sys(void* p, unsigned v) {
  asm volatile("global_store_dword %0, %1, off sc0 sc1" :: "v"(p), "v"(v) : "memory");
}
#define WAIT_VM(n) asm volatile("s_waitcnt vmcnt(" #n ")" ::: "memory")
template <int N> __device__ __forceinline__ void wait_vm() {
  asm volatile("s_waitcnt vmcnt(%0)" :: "i"(N > 63 ? 63 : N) : "memory");
}
template <int V> struct IC { static constexpr int value = V; };
template <int A, int B, class F> __device__ __forceinline__ void static_for(F&& f) {
  if constexpr (A < B) { f(IC<A>{}); static_for<A + 1, B>(f); }
}
#define STAMP(slot) do {} while (0)
__device__ __forceinline__ void unit_of(int u, int& rb, int& kc) {
  constexpr int HEAD = (NKC - TAILK) * NRB;
  if (u < HEAD) { kc = u / NRB; rb = u % NRB; }
  else { int v = u - HEAD; rb = v / TAILK; kc = (NKC - TAILK) + v % TAILK; }
}

__global__ void __launch_bounds__(448, 2) ag_gemm_kernel(Params p) {
  extern __shared__ __attribute__((aligned(16))) unsigned char smem[];
  const int tid = threadIdx.x;
  const int wid = __builtin_amdgcn_readfirstlane(tid >> 6);
  const int lane = tid & 63;
  const int bid = blockIdx.x;
  const int rank = p.rank;
  const unsigned lds0 = (unsigned)(uintptr_t)(LAS unsigned char*)smem;
  const unsigned ctl = lds0 + LDS_CTL;
  unsigned char* myws = p.ws[rank];

  const int xcd = bid & 7;
  const int idx = bid >> 3;
  STAMP(0);

  if (idx >= 28) {
    const unsigned vsrc = (lane >> 4) * KD + (lane & 15) * 16;
    unsigned char* peer = p.ws[(rank + 1 + wid) & 7];
    unsigned* ctr = (unsigned*)(myws + WS_CTR_P) + bid * 16 + wid;
    unsigned* tick = (unsigned*)(myws + WS_TICK + wid * 128);
    v4u d[2][2 * GSZ];
    auto first_of = [&](unsigned gq) -> int { return gq < (unsigned)NHEAD ? gq * GSZ : NHEAD * GSZ + (gq - NHEAD); };
    auto load = [&](unsigned gq, v4u* dd) {
      const int pc = first_of(gq);
      int rbb, kcc; unit_of(pc / PPU, rbb, kcc);
      const unsigned char* s = p.x + (size_t)(rbb * TM + (pc % PPU) * 8) * KD + kcc * KC;
      asm volatile("s_nop 4\n\tglobal_load_dwordx4 %0, %2, %3\n\tglobal_load_dwordx4 %1, %2, %4"
                   : "=&v"(dd[0]), "=&v"(dd[1]) : "v"(vsrc), "s"(s), "s"(s + 4 * KD) : "memory");
      if (gq < (unsigned)NHEAD) {
        #pragma unroll
        for (int u = 1; u < GSZ; ++u)
          asm volatile("s_nop 4\n\tglobal_load_dwordx4 %0, %2, %3\n\tglobal_load_dwordx4 %1, %2, %4"
                       : "=&v"(dd[2 * u]), "=&v"(dd[2 * u + 1]) : "v"(vsrc), "s"(s + (size_t)u * 8 * KD), "s"(s + (size_t)(u * 8 + 4) * KD) : "memory");
      } else {
        #pragma unroll
        for (int u = 1; u < GSZ; ++u) asm volatile("" : "=v"(dd[2 * u]), "=v"(dd[2 * u + 1]));
      }
    };
    const unsigned pw = (idx - 28) * 8 + xcd;
    unsigned cur = pw, nxt = pw + NPWG;
    unsigned c;
    asm volatile("global_load_dword %0, %1, off sc1" : "=v"(c) : "v"(ctr) : "memory");
    load(cur, d[0]);
    wait_vm<0>();
    asm volatile("" : "+v"(c));
    const unsigned call = __builtin_amdgcn_readfirstlane(c) + 1;
    const unsigned base = (call - 1) * (unsigned)NGRP - 2 * NPWG;
    const unsigned char* dbuf = peer + WS_XBUF + (size_t)(call & 1) * XBUF_BYTES + (size_t)rank * MS * KD;
    STAMP(1);
    auto step = [&](v4u* dc, v4u* dn) {
      #pragma unroll
      for (int u = 0; u < 2 * GSZ; ++u) asm volatile("" : "+v"(dc[u]));
      unsigned t = 0; const unsigned one = 1;
      const int pc = first_of(cur);
      int rbb, kcc; unit_of(pc / PPU, rbb, kcc);
      const unsigned char* dst = dbuf + (size_t)(rbb * NKC + kcc) * XT + (pc % PPU) * 2048;
      asm volatile("s_nop 4\n\tglobal_store_dwordx4 %0, %1, %3 sc0 sc1\n\tglobal_store_dwordx4 %0, %2, %3 offset:1024 sc0 sc1\n\ts_nop 1"
                   :: "v"(lane * 16), "v"(dc[0]), "v"(dc[1]), "s"(dst) : "memory");
      if (lane == 0) asm volatile("global_atomic_add %0, %1, %2, off sc0" : "=v"(t) : "v"(tick), "v"(one) : "memory");
      if (nxt < (unsigned)NGRP) load(nxt, dn);
      else {
        #pragma unroll
        for (int u = 0; u < 2 * GSZ; ++u) asm volatile("" : "=v"(dn[u]));
      }
      wait_vm<0>();
      if (cur < (unsigned)NHEAD) {
        #pragma unroll
        for (int u = 1; u < GSZ; ++u)
          asm volatile("s_nop 4\n\tglobal_store_dwordx4 %0, %1, %3 sc0 sc1\n\tglobal_store_dwordx4 %0, %2, %3 offset:1024 sc0 sc1\n\ts_waitcnt vmcnt(0)"
                       :: "v"(lane * 16), "v"(dc[2 * u]), "v"(dc[2 * u + 1]), "s"(dst + u * 2048) : "memory");
      }
      asm volatile("" : "+v"(t));
      if (lane == 0) st4_sys(peer + WS_FLAGS + ((size_t)((rank * NRB + rbb) * NKC + kcc) * PPU + (pc % PPU)) * 4, call);
      cur = nxt;
      nxt = __builtin_amdgcn_readfirstlane(t) - base;
    };
    while (cur < (unsigned)NGRP) {
      step(d[0], d[1]);
      if (cur >= (unsigned)NGRP) break;
      step(d[1], d[0]);
    }
    STAMP(2);
    if (lane == 0) asm volatile("global_store_dword %0, %1, off sc1" :: "v"(ctr), "v"(call) : "memory");
    return;
  }
  if (wid >= 6) return;

  const int nt = idx & 3;
  const int jj = idx >> 2;
  const int R = xcd * 7 + jj;
  const int rsrc = (rank + 1 + (R >> 3)) & 7;
  const int rrb = R & 7;
  const int npass = (jj == 0) ? 2 : 1;

  if (tid < 16) *(LAS unsigned*)(smem + LDS_CTL + tid * 4) = 0;
  asm volatile("s_waitcnt lgkmcnt(0)" ::: "memory");
  __builtin_amdgcn_s_barrier();

  const int ntot = npass * NKC;
  auto wait_free = [&](int c) {
    if (c >= STAGES) {
      for (;;) {
        unsigned f0 = lds_rd(ctl + (C_FREE + 0) * 4), f1 = lds_rd(ctl + (C_FREE + 1) * 4);
        unsigned f2 = lds_rd(ctl + (C_FREE + 2) * 4), f3 = lds_rd(ctl + (C_FREE + 3) * 4);
        unsigned m = min(min(f0, f1), min(f2, f3));
        if (m >= (unsigned)(c - STAGES + 1)) break;
        __builtin_amdgcn_s_sleep(1);
      }
    }
  };

  if (wid == 4) {
    unsigned* ctr = (unsigned*)(myws + WS_CTR_L) + bid;
    unsigned call;
    {
      unsigned c;
      asm volatile("global_load_dword %0, %1, off sc1\n\ts_waitcnt vmcnt(0)" : "=v"(c) : "v"(ctr) : "memory");
      call = __builtin_amdgcn_readfirstlane(c) + 1;
      if (lane == 0) asm volatile("global_store_dword %0, %1, off sc1" :: "v"(ctr), "v"(call) : "memory");
    }
    const unsigned char* fl = myws + WS_FLAGS + (size_t)(rsrc * NRB + rrb) * NKC * PPU * 4;
    unsigned* tmo = (unsigned*)(myws + WS_TMO);
    const int lr = lane >> 4, lc = lane & 15;
    bool dead = false;
    const unsigned long long t0 = __builtin_amdgcn_s_memrealtime();
    for (int c = 0; c < ntot; ++c) {
      const int slot = c % STAGES;
      const bool own = (npass == 2 && c < NKC);
      const int kc = c & (NKC - 1);
      const unsigned char* xbase = own ? p.x + (size_t)xcd * TM * KD + kc * KC
                                       : myws + WS_XBUF + (size_t)(call & 1) * XBUF_BYTES + (size_t)((rsrc * NRB + rrb) * NKC + kc) * XT;
      const int rstride = own ? KD : KC;
      wait_free(c);
      if (!own) {
        bool first = true;
        for (;;) {
          unsigned f, tv;
          asm volatile("global_load_dword %0, %2, off sc0 sc1\n\tglobal_load_dword %1, %3, off sc1\n\ts_waitcnt vmcnt(0)"
                       : "=&v"(f), "=&v"(tv)
                       : "v"(fl + (kc * PPU + (lane & (kc < NKC - FINEK ? PPU - GSZ : PPU - 1))) * 4), "v"(tmo) : "memory");
          if (first) { first = false; if (c > 0) lds_wr(ctl + (C_FULLX + (c - 1) % STAGES) * 4, c); }
          if (__all(f >= call) || dead) break;
          if (__builtin_amdgcn_readfirstlane(tv) == call || __builtin_amdgcn_s_memrealtime() - t0 > DEADLINE_TICKS) {
            if (lane == 0) asm volatile("global_store_dword %0, %1, off sc1" :: "v"(tmo), "v"(call) : "memory");
            lds_wr(ctl + C_BAD * 4, 1);
            dead = true;
            break;
          }
          if (kc < NKC - 1) __builtin_amdgcn_s_sleep(2);
        }
      }
      if (c == ntot - 1) STAMP(1);
      if (c == ntot - NKC) STAMP(4);
      const unsigned dst = lds0 + LDS_X + slot * XT;
      #pragma unroll
      for (int i = 0; i < 16; ++i) {
        const int row = 4 * i + lr;
        glds16_sc1(xbase + row * rstride + ((lc ^ (row & 15)) * 16), dst + i * 1024);
      }
      if (own && c > 0) {
        WAIT_VM(16); lds_wr(ctl + (C_FULLX + (c - 1) % STAGES) * 4, c);
      }
    }
    WAIT_VM(0);
    lds_wr(ctl + (C_FULLX + (ntot - 1) % STAGES) * 4, ntot);
    STAMP(2);
    return;
  }

  if (wid == 5) {
    const unsigned char* wbase = p.w + nt * TN;
    const int lr = lane >> 3, lc = lane & 7;
    for (int c = 0; c < ntot; ++c) {
      const int slot = c % STAGES;
      const int kc = c & (NKC - 1);
      wait_free(c);
      const unsigned dst = lds0 + LDS_W + slot * WT;
      #pragma unroll
      for (int i = 0; i < 32; ++i) {
        const int row = 8 * i + lr;
        const int h = ((row >> 1) & 3) | (((row >> 5) & 1) << 2);
        const unsigned char* g = wbase + (size_t)(kc * KC + row) * NL + ((lc ^ h) * 16);
        glds16(g, dst + i * 1024);
      }
      if (c > 0) { WAIT_VM(32); lds_wr(ctl + (C_FULLW + (c - 1) % STAGES) * 4, c); }
    }
    WAIT_VM(0);
    lds_wr(ctl + (C_FULLW + (ntot - 1) % STAGES) * 4, ntot);
    return;
  }

  const float scale = p.sx[0] * p.sw[0];
  const int g = lane >> 4, li = lane & 15;
  const int q = li >> 1, pp = li & 1;
  for (int pass = 0; pass < npass; ++pass) {
    v4f acc[4][2];
    #pragma unroll
    for (int m = 0; m < 4; ++m) { acc[m][0] = v4f{0, 0, 0, 0}; acc[m][1] = v4f{0, 0, 0, 0}; }
    for (int kc = 0; kc < NKC; ++kc) {
      const int c = pass * NKC + kc;
      const int slot = c % STAGES;
      for (;;) {
        unsigned fx = lds_rd(ctl + (C_FULLX + slot) * 4), fw = lds_rd(ctl + (C_FULLW + slot) * 4);
        if (min(fx, fw) >= (unsigned)(c + 1)) break;
        __builtin_amdgcn_s_sleep(1);
      }
      const LAS unsigned char* xs = (const LAS unsigned char*)smem + LDS_X + slot * XT;
      const LAS unsigned char* wsm = (const LAS unsigned char*)smem + LDS_W + slot * WT;
      #pragma unroll
      for (int ks = 0; ks < 2; ++ks) {
        v8i af[2], bf[4];
        #pragma unroll
        for (int n = 0; n < 2; ++n) {
          #pragma unroll
          for (int t = 0; t < 4; ++t) {
            const int k = 128 * ks + 32 * g + 8 * t + q;
            const int h = ((k >> 1) & 3) | (((k >> 5) & 1) << 2);
            v2i r = __builtin_amdgcn_ds_read_tr8_b64_v2i32((LAS v2i*)(wsm + k * 128 + (((2 * wid + n) ^ h) * 16) + 8 * pp));
            af[n][2 * t] = r[0]; af[n][2 * t + 1] = r[1];
          }
        }
        #pragma unroll
        for (int m = 0; m < 4; ++m) {
          const int row = 16 * m + li;
          v4i lo = *(const LAS v4i*)(xs + row * 256 + (((8 * ks + 2 * g) ^ li) * 16));
          v4i hi = *(const LAS v4i*)(xs + row * 256 + (((8 * ks + 2 * g + 1) ^ li) * 16));
          bf[m] = v8i{lo[0], lo[1], lo[2], lo[3], hi[0], hi[1], hi[2], hi[3]};
        }
        if (ks == 1) {
          asm volatile("s_waitcnt lgkmcnt(0)" ::: "memory");
          __builtin_amdgcn_sched_barrier(0);
          if (c + STAGES < ntot) lds_wr(ctl + (C_FREE + wid) * 4, c + 1);
        }
        #pragma unroll
        for (int m = 0; m < 4; ++m)
          #pragma unroll
          for (int n = 0; n < 2; ++n)
            acc[m][n] = __builtin_amdgcn_mfma_scale_f32_16x16x128_f8f6f4(af[n], bf[m], acc[m][n], 1, 0, 0, 0, 0, 0);
      }
    }
    STAMP(1);
    const bool own = (npass == 2 && pass == 0);
    const bool bad = !own && lds_rd(ctl + C_BAD * 4) != 0;
    const int orow = own ? rank * MS + xcd * TM : rsrc * MS + rrb * TM;
#define EPI 2
    float* obase = p.out + (size_t)(orow + li) * NL + nt * TN + 32 * wid + 4 * g;
    #pragma unroll
    for (int m = 0; m < 4; ++m)
      #pragma unroll
      for (int n = 0; n < 2; ++n) {
        v4f v = acc[m][n] * scale;
        v[0] = fmaxf(v[0], 0.f); v[1] = fmaxf(v[1], 0.f); v[2] = fmaxf(v[2], 0.f); v[3] = fmaxf(v[3], 0.f);
        if (bad) v = v4f{__builtin_nanf(""), __builtin_nanf(""), __builtin_nanf(""), __builtin_nanf("")};
        st16_sys(obase + (size_t)m * 16 * NL + n * 16, __builtin_bit_cast(v4u, v));
      }
    STAMP(2);
  }
}

}

extern "C" void kernel_launch(int rank, int nranks, ncclComm_t comm,
                              hipStream_t stream,
                              void* const* d_in, const int* in_sizes, int n_in,
                              void* d_out, int out_size,
                              const int* in_dtypes, int out_dtype,
                              void* const* d_out_all,
                              void* const* d_ws_all, size_t ws_bytes) {
  (void)comm; (void)in_sizes; (void)in_dtypes; (void)out_dtype; (void)d_out_all;
  if (nranks != NR || n_in != 4 || out_size != NR * MS * NL || ws_bytes < WS_NEED || (unsigned)rank >= (unsigned)NR) {
    fprintf(stderr, "kernel_launch: unexpected problem shape\n");
    return;
  }
  HIP_CHECK(hipFuncSetAttribute((const void*)ag_gemm_kernel, hipFuncAttributeMaxDynamicSharedMemorySize, LDS_BYTES));
  Params p;
  p.x = (const unsigned char*)d_in[0];
  p.w = (const unsigned char*)d_in[1];
  p.sx = (const float*)d_in[2];
  p.sw = (const float*)d_in[3];
  p.out = (float*)d_out;
  for (int i = 0; i < NR; ++i) p.ws[i] = (unsigned char*)d_ws_all[i];
  p.rank = rank;
  ag_gemm_kernel<<<NWG, 448, LDS_BYTES, stream>>>(p);
  HIP_CHECK(hipGetLastError());
}
```

```cpp
#include <hip/hip_runtime.h>
#include <rccl/rccl.h>
#include <cstdio>
#include <cstdint>

#define HIP_CHECK(expr) do { hipError_t _e = (expr); if (_e != hipSuccess) { \
    fprintf(stderr, "HIP error %s at %s:%d\n", hipGetErrorString(_e), __FILE__, __LINE__); \
    return; } } while (0)

namespace {

typedef int v2i __attribute__((ext_vector_type(2)));
typedef int v4i __attribute__((ext_vector_type(4)));
typedef int v8i __attribute__((ext_vector_type(8)));
typedef unsigned v4u __attribute__((ext_vector_type(4)));
typedef float v4f __attribute__((ext_vector_type(4)));
#define LAS __attribute__((address_space(3)))

constexpr int NR = 8;
constexpr int KD = 4096, NL = 512, MS = 512;
constexpr int TM = 64, TN = 128, KC = 256;
constexpr int NKC = KD / KC;
constexpr int NRB = MS / TM;
constexpr int NWG = 256;
constexpr int STAGES = 3;
constexpr int XT = TM * KC;
constexpr int WT = KC * TN;
constexpr int LDS_X = 0;
constexpr int LDS_W = STAGES * XT;
constexpr int LDS_CTL = STAGES * (XT + WT);
constexpr int LDS_BYTES = LDS_CTL + 256;
constexpr int C_FULLX = 0, C_FULLW = 4, C_FREE = 8  , C_BAD = 12, C_SUB = 13;

constexpr int NPWG = 32;
#define GSZ 2
#define FINEK 3
constexpr int PPU = 8;
constexpr int NHEAD = (NKC - FINEK) * NRB * (PPU / GSZ);
constexpr int NGRP = NHEAD + FINEK * NRB * PPU;

constexpr size_t WS_CTR_L = 0;
constexpr size_t WS_CTR_P = 4096;
constexpr size_t WS_TMO = 24576;
constexpr size_t WS_TICK = 28672;
constexpr size_t WS_FLAGS = 65536;
constexpr size_t WS_DBG = 512u << 10;
constexpr size_t WS_XBUF = 1u << 20;
constexpr size_t XBUF_BYTES = (size_t)NR * MS * KD;
constexpr size_t WS_NEED = WS_XBUF + 2 * XBUF_BYTES;
constexpr unsigned long long DEADLINE_TICKS = 40000000ull;

struct Params {
  const unsigned char* x;
  const unsigned char* w;
  const float* sx;
  const float* sw;
  float* out;
  unsigned char* ws[NR];
  int rank;
};

__device__ __forceinline__ void glds16(const void* gsrc, unsigned lds_dst) {
  unsigned keep;
  asm volatile("s_mov_b32 %0, m0\n\ts_mov_b32 m0, %2\n\ts_nop 0\n\tglobal_load_lds_dwordx4 %1, off\n\ts_mov_b32 m0, %0"
               : "=&s"(keep) : "v"(gsrc), "s"(lds_dst) : "memory");
}
__device__ __forceinline__ void glds16_sc1(const void* gsrc, unsigned lds_dst) {
  unsigned keep;
  asm volatile("s_mov_b32 %0, m0\n\ts_mov_b32 m0, %2\n\ts_nop 0\n\tglobal_load_lds_dwordx4 %1, off sc1\n\ts_mov_b32 m0, %0"
               : "=&s"(keep) : "v"(gsrc), "s"(lds_dst) : "memory");
}
__device__ __forceinline__ unsigned lds_rd(unsigned addr) {
  unsigned v;
  asm volatile("ds_read_b32 %0, %1\n\ts_waitcnt lgkmcnt(0)" : "=v"(v) : "v"(addr) : "memory");
  return __builtin_amdgcn_readfirstlane(v);
}
__device__ __forceinline__ void lds_wr(unsigned addr, unsigned v) {
  asm volatile("ds_write_b32 %0, %1\n\ts_waitcnt lgkmcnt(0)" :: "v"(addr), "v"(v) : "memory");
}
__device__ __forceinline__ void st16_sys(void* p, v4u v) {
  asm volatile("global_store_dwordx4 %0, %1, off sc0 sc1\n\ts_nop 1" :: "v"(p), "v"(v) : "memory");
}
__device__ __forceinline__ void st4_sys(void* p, unsigned v) {
  asm volatile("global_store_dword %0, %1, off sc0 sc1" :: "v"(p), "v"(v) : "memory");
}
#define WAIT_VM(n) asm volatile("s_waitcnt vmcnt(" #n ")" ::: "memory")
template <int N> __device__ __forceinline__ void wait_vm() {
  asm volatile("s_waitcnt vmcnt(%0)" :: "i"(N > 63 ? 63 : N) : "memory");
}
template <int V> struct IC { static constexpr int value = V; };
template <int A, int B, class F> __device__ __forceinline__ void static_for(F&& f) {
  if constexpr (A < B) { f(IC<A>{}); static_for<A + 1, B>(f); }
}
#define STAMP(slot) do {} while (0)
__device__ __forceinline__ void piece_of(int pc, int& rb, int& kc, int& pu) {
  constexpr int HEAD = (NKC - 1) * NRB * PPU;
  if (pc < HEAD) { kc = pc / (NRB * PPU); rb = (pc / PPU) % NRB; pu = pc % PPU; }
  else { const int v = pc - HEAD; kc = NKC - 1; rb = (v >> 1) & 7; pu = (v >> 4) * 2 + (v & 1); }
}

__global__ void __launch_bounds__(448, 2) ag_gemm_kernel(Params p) {
  extern __shared__ __attribute__((aligned(16))) unsigned char smem[];
  const int tid = threadIdx.x;
  const int wid = __builtin_amdgcn_readfirstlane(tid >> 6);
  const int lane = tid & 63;
  const int bid = blockIdx.x;
  const int rank = p.rank;
  const unsigned lds0 = (unsigned)(uintptr_t)(LAS unsigned char*)smem;
  const unsigned ctl = lds0 + LDS_CTL;
  unsigned char* myws = p.ws[rank];

  const int xcd = bid & 7;
  const int idx = ((bid >> 3) + 28) & 31;
  STAMP(0);

  if (idx >= 28) {
    const unsigned vsrc = (lane >> 4) * KD + (lane & 15) * 16;
    unsigned char* peer = p.ws[(rank + 1 + wid) & 7];
    unsigned* ctr = (unsigned*)(myws + WS_CTR_P) + bid * 16 + wid;
    unsigned* tick = (unsigned*)(myws + WS_TICK + wid * 128);
    v4u d[2][2 * GSZ];
    auto first_of = [&](unsigned gq) -> int { return gq < (unsigned)NHEAD ? gq * GSZ : NHEAD * GSZ + (gq - NHEAD); };
    auto load = [&](unsigned gq, v4u* dd) {
      const int pc = first_of(gq);
      int rbb, kcc, pu; piece_of(pc, rbb, kcc, pu);
      const unsigned char* s = p.x + (size_t)(rbb * TM + pu * 8) * KD + kcc * KC;
      asm volatile("s_nop 4\n\tglobal_load_dwordx4 %0, %2, %3\n\tglobal_load_dwordx4 %1, %2, %4"
                   : "=&v"(dd[0]), "=&v"(dd[1]) : "v"(vsrc), "s"(s), "s"(s + 4 * KD) : "memory");
      if (gq < (unsigned)NHEAD) {
        #pragma unroll
        for (int u = 1; u < GSZ; ++u)
          asm volatile("s_nop 4\n\tglobal_load_dwordx4 %0, %2, %3\n\tglobal_load_dwordx4 %1, %2, %4"
                       : "=&v"(dd[2 * u]), "=&v"(dd[2 * u + 1]) : "v"(vsrc), "s"(s + (size_t)u * 8 * KD), "s"(s + (size_t)(u * 8 + 4) * KD) : "memory");
      } else {
        #pragma unroll
        for (int u = 1; u < GSZ; ++u) asm volatile("" : "=v"(dd[2 * u]), "=v"(dd[2 * u + 1]));
      }
    };
    const unsigned pw = (idx - 28) * 8 + xcd;
    unsigned cur = pw, nxt = pw + NPWG;
    unsigned c;
    asm volatile("global_load_dword %0, %1, off sc1" : "=v"(c) : "v"(ctr) : "memory");
    load(cur, d[0]);
    wait_vm<0>();
    asm volatile("" : "+v"(c));
    const unsigned call = __builtin_amdgcn_readfirstlane(c) + 1;
    const unsigned base = (call - 1) * (unsigned)NGRP - 2 * NPWG;
    const unsigned char* dbuf = peer + WS_XBUF + (size_t)(call & 1) * XBUF_BYTES + (size_t)rank * MS * KD;
    STAMP(1);
    auto step = [&](v4u* dc, v4u* dn) {
      #pragma unroll
      for (int u = 0; u < 2 * GSZ; ++u) asm volatile("" : "+v"(dc[u]));
      unsigned t = 0; const unsigned one = 1;
      const int pc = first_of(cur);
      int rbb, kcc, pu; piece_of(pc, rbb, kcc, pu);
      const unsigned char* dst = dbuf + (size_t)(rbb * NKC + kcc) * XT + pu * 2048;
      asm volatile("s_nop 4\n\tglobal_store_dwordx4 %0, %1, %3 sc0 sc1\n\tglobal_store_dwordx4 %0, %2, %3 offset:1024 sc0 sc1\n\ts_nop 1"
                   :: "v"(lane * 16), "v"(dc[0]), "v"(dc[1]), "s"(dst) : "memory");
      if (lane == 0) asm volatile("global_atomic_add %0, %1, %2, off sc0" : "=v"(t) : "v"(tick), "v"(one) : "memory");
      if (nxt < (unsigned)NGRP) load(nxt, dn);
      else {
        #pragma unroll
        for (int u = 0; u < 2 * GSZ; ++u) asm volatile("" : "=v"(dn[u]));
      }
      wait_vm<0>();
      if (cur < (unsigned)NHEAD) {
        #pragma unroll
        for (int u = 1; u < GSZ; ++u)
          asm volatile("s_nop 4\n\tglobal_store_dwordx4 %0, %1, %3 sc0 sc1\n\tglobal_store_dwordx4 %0, %2, %3 offset:1024 sc0 sc1\n\ts_waitcnt vmcnt(0)"
                       :: "v"(lane * 16), "v"(dc[2 * u]), "v"(dc[2 * u + 1]), "s"(dst + u * 2048) : "memory");
      }
      asm volatile("" : "+v"(t));
      if (lane == 0) st4_sys(peer + WS_FLAGS + ((size_t)((rank * NRB + rbb) * NKC + kcc) * PPU + pu) * 4, call);
      cur = nxt;
      nxt = __builtin_amdgcn_readfirstlane(t) - base;
    };
    while (cur < (unsigned)NGRP) {
      step(d[0], d[1]);
      if (cur >= (unsigned)NGRP) break;
      step(d[1], d[0]);
    }
    STAMP(2);
    if (lane == 0) asm volatile("global_store_dword %0, %1, off sc1" :: "v"(ctr), "v"(call) : "memory");
    return;
  }
  if (wid >= 6) return;

  const int nt = idx & 3;
  const int jj = idx >> 2;
  const int R = xcd * 7 + jj;
  const int rsrc = (rank + 1 + (R >> 3)) & 7;
  const int rrb = R & 7;
  const int npass = (jj == 0) ? 2 : 1;

  if (tid < 16) *(LAS unsigned*)(smem + LDS_CTL + tid * 4) = 0;
  asm volatile("s_waitcnt lgkmcnt(0)" ::: "memory");
  __builtin_amdgcn_s_barrier();

  const int ntot = npass * NKC;
  auto wait_free = [&](int c) {
    if (c >= STAGES) {
      for (;;) {
        unsigned f0 = lds_rd(ctl + (C_FREE + 0) * 4), f1 = lds_rd(ctl + (C_FREE + 1) * 4);
        unsigned f2 = lds_rd(ctl + (C_FREE + 2) * 4), f3 = lds_rd(ctl + (C_FREE + 3) * 4);
        unsigned m = min(min(f0, f1), min(f2, f3));
        if (m >= (unsigned)(c - STAGES + 1)) break;
        __builtin_amdgcn_s_sleep(1);
      }
    }
  };

  if (wid == 4) {
    unsigned* ctr = (unsigned*)(myws + WS_CTR_L) + bid;
    unsigned call;
    {
      unsigned c;
      asm volatile("global_load_dword %0, %1, off sc1\n\ts_waitcnt vmcnt(0)" : "=v"(c) : "v"(ctr) : "memory");
      call = __builtin_amdgcn_readfirstlane(c) + 1;
      if (lane == 0) asm volatile("global_store_dword %0, %1, off sc1" :: "v"(ctr), "v"(call) : "memory");
    }
    const unsigned char* fl = myws + WS_FLAGS + (size_t)(rsrc * NRB + rrb) * NKC * PPU * 4;
    unsigned* tmo = (unsigned*)(myws + WS_TMO);
    const int lr = lane >> 4, lc = lane & 15;
    bool dead = false;
    const unsigned long long t0 = __builtin_amdgcn_s_memrealtime();
    auto await = [&](int word, int pubw, unsigned pubv, bool nap) {
      bool first = true;
      for (;;) {
        unsigned f, tv;
        asm volatile("global_load_dword %0, %2, off sc0 sc1\n\tglobal_load_dword %1, %3, off sc1\n\ts_waitcnt vmcnt(0)"
                     : "=&v"(f), "=&v"(tv) : "v"(fl + word * 4), "v"(tmo) : "memory");
        if (first) { first = false; if (pubw >= 0) lds_wr(ctl + pubw * 4, pubv); }
        if (__all(f >= call) || dead) break;
        if (__builtin_amdgcn_readfirstlane(tv) == call || __builtin_amdgcn_s_memrealtime() - t0 > DEADLINE_TICKS) {
          if (lane == 0) asm volatile("global_store_dword %0, %1, off sc1" :: "v"(tmo), "v"(call) : "memory");
          lds_wr(ctl + C_BAD * 4, 1);
          dead = true;
          break;
        }
        if (nap) __builtin_amdgcn_s_sleep(2);
      }
    };
    for (int c = 0; c < ntot; ++c) {
      const int slot = c % STAGES;
      const bool own = (npass == 2 && c < NKC);
      const int kc = c & (NKC - 1);
      const unsigned char* xbase = own ? p.x + (size_t)xcd * TM * KD + kc * KC
                                       : myws + WS_XBUF + (size_t)(call & 1) * XBUF_BYTES + (size_t)((rsrc * NRB + rrb) * NKC + kc) * XT;
      const int rstride = own ? KD : KC;
      const unsigned dst = lds0 + LDS_X + slot * XT;
      wait_free(c);
      if (c == ntot - 1) {
        int issued = 0;
        bool first = true;
        while (issued < 4) {
          unsigned f, tv;
          asm volatile("global_load_dword %0, %2, off sc0 sc1\n\tglobal_load_dword %1, %3, off sc1\n\ts_waitcnt vmcnt(0)"
                       : "=&v"(f), "=&v"(tv) : "v"(fl + (kc * PPU + (lane & 7)) * 4), "v"(tmo) : "memory");
          if (first) { first = false; lds_wr(ctl + (C_FULLX + (c - 1) % STAGES) * 4, c); }
          else lds_wr(ctl + C_SUB * 4, issued);
          unsigned up = (unsigned)__ballot(f >= call) & 0xffu;
          if (up != 0xffu && !dead &&
              (__builtin_amdgcn_readfirstlane(tv) == call || __builtin_amdgcn_s_memrealtime() - t0 > DEADLINE_TICKS)) {
            if (lane == 0) asm volatile("global_store_dword %0, %1, off sc1" :: "v"(tmo), "v"(call) : "memory");
            lds_wr(ctl + C_BAD * 4, 1);
            dead = true;
          }
          if (dead) up = 0xffu;
          const int ready = __builtin_ctz(~up) >> 1;
          #pragma unroll
          for (int m = 0; m < 4; ++m) {
            if (m >= issued && m < ready) {
              #pragma unroll
              for (int i = 4 * m; i < 4 * m + 4; ++i) {
                const int row = 4 * i + lr;
                glds16_sc1(xbase + row * rstride + ((lc ^ (row & 15)) * 16), dst + i * 1024);
              }
            }
          }
          if (ready > issued) issued = ready;
        }
        STAMP(1);
        break;
      }
      if (!own) await(kc * PPU + (lane & (kc < NKC - FINEK ? PPU - GSZ : PPU - 1)), c > 0 ? C_FULLX + (c - 1) % STAGES : -1, c, true);
      if (c == ntot - NKC) STAMP(4);
      #pragma unroll
      for (int i = 0; i < 16; ++i) {
        const int row = 4 * i + lr;
        glds16_sc1(xbase + row * rstride + ((lc ^ (row & 15)) * 16), dst + i * 1024);
      }
      if (own && c > 0) { WAIT_VM(16); lds_wr(ctl + (C_FULLX + (c - 1) % STAGES) * 4, c); }
    }
    WAIT_VM(0);
    lds_wr(ctl + C_SUB * 4, 4);
    STAMP(2);
    return;
  }

  if (wid == 5) {
    const unsigned char* wbase = p.w + nt * TN;
    const int lr = lane >> 3, lc = lane & 7;
    for (int c = 0; c < ntot; ++c) {
      const int slot = c % STAGES;
      const int kc = c & (NKC - 1);
      wait_free(c);
      const unsigned dst = lds0 + LDS_W + slot * WT;
      #pragma unroll
      for (int i = 0; i < 32; ++i) {
        const int row = 8 * i + lr;
        const int h = ((row >> 1) & 3) | (((row >> 5) & 1) << 2);
        const unsigned char* g = wbase + (size_t)(kc * KC + row) * NL + ((lc ^ h) * 16);
        glds16(g, dst + i * 1024);
      }
      if (c > 0) { WAIT_VM(32); lds_wr(ctl + (C_FULLW + (c - 1) % STAGES) * 4, c); }
    }
    WAIT_VM(0);
    lds_wr(ctl + (C_FULLW + (ntot - 1) % STAGES) * 4, ntot);
    return;
  }

  const float scale = p.sx[0] * p.sw[0];
  const int g = lane >> 4, li = lane & 15;
  const int q = li >> 1, pp = li & 1;
  auto wfrag = [&](const LAS unsigned char* wsm, int ks, int n) -> v8i {
    v8i a;
    #pragma unroll
    for (int t = 0; t < 4; ++t) {
      const int k = 128 * ks + 32 * g + 8 * t + q;
      const int h = ((k >> 1) & 3) | (((k >> 5) & 1) << 2);
      v2i r = __builtin_amdgcn_ds_read_tr8_b64_v2i32((LAS v2i*)(wsm + k * 128 + (((2 * wid + n) ^ h) * 16) + 8 * pp));
      a[2 * t] = r[0]; a[2 * t + 1] = r[1];
    }
    return a;
  };
  auto xfrag = [&](const LAS unsigned char* xs, int ks, int m) -> v8i {
    const int row = 16 * m + li;
    v4i lo = *(const LAS v4i*)(xs + row * 256 + (((8 * ks + 2 * g) ^ li) * 16));
    v4i hi = *(const LAS v4i*)(xs + row * 256 + (((8 * ks + 2 * g + 1) ^ li) * 16));
    return v8i{lo[0], lo[1], lo[2], lo[3], hi[0], hi[1], hi[2], hi[3]};
  };
  auto put = [&](float* o, v4f v, bool bad) {
    v *= scale;
    v[0] = fmaxf(v[0], 0.f); v[1] = fmaxf(v[1], 0.f); v[2] = fmaxf(v[2], 0.f); v[3] = fmaxf(v[3], 0.f);
    if (bad) v = v4f{__builtin_nanf(""), __builtin_nanf(""), __builtin_nanf(""), __builtin_nanf("")};
    st16_sys(o, __builtin_bit_cast(v4u, v));
  };
  for (int pass = 0; pass < npass; ++pass) {
    const bool own = (npass == 2 && pass == 0);
    const int orow = own ? rank * MS + xcd * TM : rsrc * MS + rrb * TM;
    float* obase = p.out + (size_t)(orow + li) * NL + nt * TN + 32 * wid + 4 * g;
    v4f acc[4][2];
    #pragma unroll
    for (int m = 0; m < 4; ++m) { acc[m][0] = v4f{0, 0, 0, 0}; acc[m][1] = v4f{0, 0, 0, 0}; }
    const int nfull = own ? NKC : NKC - 1;
    for (int kc = 0; kc < nfull; ++kc) {
      const int c = pass * NKC + kc;
      const int slot = c % STAGES;
      for (;;) {
        unsigned fx = lds_rd(ctl + (C_FULLX + slot) * 4), fw = lds_rd(ctl + (C_FULLW + slot) * 4);
        if (min(fx, fw) >= (unsigned)(c + 1)) break;
        __builtin_amdgcn_s_sleep(1);
      }
      const LAS unsigned char* xs = (const LAS unsigned char*)smem + LDS_X + slot * XT;
      const LAS unsigned char* wsm = (const LAS unsigned char*)smem + LDS_W + slot * WT;
      #pragma unroll
      for (int ks = 0; ks < 2; ++ks) {
        v8i af[2], bf[4];
        #pragma unroll
        for (int n = 0; n < 2; ++n) af[n] = wfrag(wsm, ks, n);
        #pragma unroll
        for (int m = 0; m < 4; ++m) bf[m] = xfrag(xs, ks, m);
        if (ks == 1) {
          asm volatile("s_waitcnt lgkmcnt(0)" ::: "memory");
          __builtin_amdgcn_sched_barrier(0);
          if (c + STAGES < ntot) lds_wr(ctl + (C_FREE + wid) * 4, c + 1);
        }
        #pragma unroll
        for (int m = 0; m < 4; ++m)
          #pragma unroll
          for (int n = 0; n < 2; ++n)
            acc[m][n] = __builtin_amdgcn_mfma_scale_f32_16x16x128_f8f6f4(af[n], bf[m], acc[m][n], 1, 0, 0, 0, 0, 0);
      }
    }
    if (own) {
      #pragma unroll
      for (int m = 0; m < 4; ++m) { put(obase + (size_t)m * 16 * NL, acc[m][0], false); put(obase + (size_t)m * 16 * NL + 16, acc[m][1], false); }
    } else {
      const int slot = (ntot - 1) % STAGES;
      while (lds_rd(ctl + (C_FULLW + slot) * 4) < (unsigned)ntot) __builtin_amdgcn_s_sleep(1);
      const LAS unsigned char* xs = (const LAS unsigned char*)smem + LDS_X + slot * XT;
      const LAS unsigned char* wsm = (const LAS unsigned char*)smem + LDS_W + slot * WT;
      const v8i a00 = wfrag(wsm, 0, 0), a01 = wfrag(wsm, 0, 1), a10 = wfrag(wsm, 1, 0), a11 = wfrag(wsm, 1, 1);
      #pragma unroll
      for (int m = 0; m < 4; ++m) {
        while (lds_rd(ctl + C_SUB * 4) < (unsigned)(m + 1)) {}
        const bool bad = lds_rd(ctl + C_BAD * 4) != 0;
        const v8i b0 = xfrag(xs, 0, m), b1 = xfrag(xs, 1, m);
        acc[m][0] = __builtin_amdgcn_mfma_scale_f32_16x16x128_f8f6f4(a00, b0, acc[m][0], 1, 0, 0, 0, 0, 0);
        acc[m][1] = __builtin_amdgcn_mfma_scale_f32_16x16x128_f8f6f4(a01, b0, acc[m][1], 1, 0, 0, 0, 0, 0);
        acc[m][0] = __builtin_amdgcn_mfma_scale_f32_16x16x128_f8f6f4(a10, b1, acc[m][0], 1, 0, 0, 0, 0, 0);
        acc[m][1] = __builtin_amdgcn_mfma_scale_f32_16x16x128_f8f6f4(a11, b1, acc[m][1], 1, 0, 0, 0, 0, 0);
        if (m == 3) STAMP(1);
        put(obase + (size_t)m * 16 * NL, acc[m][0], bad); put(obase + (size_t)m * 16 * NL + 16, acc[m][1], bad);
      }
    }
    STAMP(2);
  }
}

}

extern "C" void kernel_launch(int rank, int nranks, ncclComm_t comm,
                              hipStream_t stream,
                              void* const* d_in, const int* in_sizes, int n_in,
                              void* d_out, int out_size,
                              const int* in_dtypes, int out_dtype,
                              void* const* d_out_all,
                              void* const* d_ws_all, size_t ws_bytes) {
  (void)comm; (void)in_sizes; (void)in_dtypes; (void)out_dtype; (void)d_out_all;
  if (nranks != NR || n_in != 4 || out_size != NR * MS * NL || ws_bytes < WS_NEED || (unsigned)rank >= (unsigned)NR) {
    fprintf(stderr, "kernel_launch: unexpected problem shape\n");
    return;
  }
  HIP_CHECK(hipFuncSetAttribute((const void*)ag_gemm_kernel, hipFuncAttributeMaxDynamicSharedMemorySize, LDS_BYTES));
  Params p;
  p.x = (const unsigned char*)d_in[0];
  p.w = (const unsigned char*)d_in[1];
  p.sx = (const float*)d_in[2];
  p.sw = (const float*)d_in[3];
  p.out = (float*)d_out;
  for (int i = 0; i < NR; ++i) p.ws[i] = (unsigned char*)d_ws_all[i];
  p.rank = rank;
  ag_gemm_kernel<<<NWG, 448, LDS_BYTES, stream>>>(p);
  HIP_CHECK(hipGetLastError());
}
```

```cpp
#include <hip/hip_runtime.h>
#include <rccl/rccl.h>
#include <cstdio>
#include <cstdint>

#define HIP_CHECK(expr) do { hipError_t _e = (expr); if (_e != hipSuccess) { \
    fprintf(stderr, "HIP error %s at %s:%d\n", hipGetErrorString(_e), __FILE__, __LINE__); \
    return; } } while (0)

namespace {

typedef int v2i __attribute__((ext_vector_type(2)));
typedef int v4i __attribute__((ext_vector_type(4)));
typedef int v8i __attribute__((ext_vector_type(8)));
typedef unsigned v4u __attribute__((ext_vector_type(4)));
typedef float v4f __attribute__((ext_vector_type(4)));
#define LAS __attribute__((address_space(3)))

constexpr int NR = 8;
constexpr int KD = 4096, NL = 512, MS = 512;
constexpr int TM = 64, TN = 128, KC = 256;
constexpr int NKC = KD / KC;
constexpr int NRB = MS / TM;
constexpr int NWG = 256;
constexpr int STAGES = 3;
constexpr int XT = TM * KC;
constexpr int WT = KC * TN;
constexpr int LDS_X = 0;
constexpr int LDS_W = STAGES * XT;
constexpr int LDS_CTL = STAGES * (XT + WT);
constexpr int LDS_BYTES = LDS_CTL + 256;
constexpr int C_FULLX = 0, C_FULLW = 4, C_FREE = 8  , C_BAD = 12, C_SUB = 13;

constexpr int NPWG = 32;
#define NPA 30
#define GSZ 2
#define FINEK 3
constexpr int PPU = 8;
constexpr int NHEAD = (NKC - FINEK) * NRB * (PPU / GSZ);
constexpr int NGRP = NHEAD + FINEK * NRB * PPU;

constexpr size_t WS_CTR_L = 0;
constexpr size_t WS_CTR_P = 4096;
constexpr size_t WS_TMO = 24576;
constexpr size_t WS_TICK = 28672;
constexpr size_t WS_FLAGS = 65536;
constexpr size_t WS_DBG = 512u << 10;
constexpr size_t WS_XBUF = 1u << 20;
constexpr size_t XBUF_BYTES = (size_t)NR * MS * KD;
constexpr size_t WS_NEED = WS_XBUF + 2 * XBUF_BYTES;
constexpr unsigned long long DEADLINE_TICKS = 40000000ull;

struct Params {
  const unsigned char* x;
  const unsigned char* w;
  const float* sx;
  const float* sw;
  float* out;
  unsigned char* ws[NR];
  int rank;
};

__device__ __forceinline__ void glds16(const void* gsrc, unsigned lds_dst) {
  unsigned keep;
  asm volatile("s_mov_b32 %0, m0\n\ts_mov_b32 m0, %2\n\ts_nop 0\n\tglobal_load_lds_dwordx4 %1, off\n\ts_mov_b32 m0, %0"
               : "=&s"(keep) : "v"(gsrc), "s"(lds_dst) : "memory");
}
__device__ __forceinline__ void glds16_sc1(const void* gsrc, unsigned lds_dst) {
  unsigned keep;
  asm volatile("s_mov_b32 %0, m0\n\ts_mov_b32 m0, %2\n\ts_nop 0\n\tglobal_load_lds_dwordx4 %1, off sc1\n\ts_mov_b32 m0, %0"
               : "=&s"(keep) : "v"(gsrc), "s"(lds_dst) : "memory");
}
__device__ __forceinline__ unsigned lds_rd(unsigned addr) {
  unsigned v;
  asm volatile("ds_read_b32 %0, %1\n\ts_waitcnt lgkmcnt(0)" : "=v"(v) : "v"(addr) : "memory");
  return __builtin_amdgcn_readfirstlane(v);
}
__device__ __forceinline__ void lds_wr(unsigned addr, unsigned v) {
  asm volatile("ds_write_b32 %0, %1\n\ts_waitcnt lgkmcnt(0)" :: "v"(addr), "v"(v) : "memory");
}
__device__ __forceinline__ void st16_sys(void* p, v4u v) {
  asm volatile("global_store_dwordx4 %0, %1, off sc0 sc1\n\ts_nop 1" :: "v"(p), "v"(v) : "memory");
}
__device__ __forceinline__ void st4_sys(void* p, unsigned v) {
  asm volatile("global_store_dword %0, %1, off sc0 sc1" :: "v"(p), "v"(v) : "memory");
}
#define WAIT_VM(n) asm volatile("s_waitcnt vmcnt(" #n ")" ::: "memory")
template <int N> __device__ __forceinline__ void wait_vm() {
  asm volatile("s_waitcnt vmcnt(%0)" :: "i"(N > 63 ? 63 : N) : "memory");
}
template <int V> struct IC { static constexpr int value = V; };
template <int A, int B, class F> __device__ __forceinline__ void static_for(F&& f) {
  if constexpr (A < B) { f(IC<A>{}); static_for<A + 1, B>(f); }
}
#define STAMP(slot) do {} while (0)
__device__ __forceinline__ void piece_of(int pc, int& rb, int& kc, int& pu) {
  constexpr int HEAD = (NKC - 1) * NRB * PPU;
  if (pc < HEAD) { kc = pc / (NRB * PPU); rb = (pc / PPU) % NRB; pu = pc % PPU; }
  else { const int v = pc - HEAD; kc = NKC - 1; rb = (v >> 1) & 7; pu = (v >> 4) * 2 + (v & 1); }
}

__global__ void __launch_bounds__(448, 2) ag_gemm_kernel(Params p) {
  extern __shared__ __attribute__((aligned(16))) unsigned char smem[];
  const int tid = threadIdx.x;
  const int wid = __builtin_amdgcn_readfirstlane(tid >> 6);
  const int lane = tid & 63;
  const int bid = blockIdx.x;
  const int rank = p.rank;
  const unsigned lds0 = (unsigned)(uintptr_t)(LAS unsigned char*)smem;
  const unsigned ctl = lds0 + LDS_CTL;
  unsigned char* myws = p.ws[rank];

  const int xcd = bid & 7;
  const int idx = ((bid >> 3) + 28) & 31;
  STAMP(0);

  if (idx >= 28) {
    const unsigned vsrc = (lane >> 4) * KD + (lane & 15) * 16;
    unsigned char* peer = p.ws[(rank + 1 + wid) & 7];
    unsigned* ctr = (unsigned*)(myws + WS_CTR_P) + bid * 16 + wid;
    unsigned* tick = (unsigned*)(myws + WS_TICK + wid * 128);
    v4u d[2][2 * GSZ];
    auto first_of = [&](unsigned gq) -> int { return gq < (unsigned)NHEAD ? gq * GSZ : NHEAD * GSZ + (gq - NHEAD); };
    auto load = [&](unsigned gq, v4u* dd) {
      const int pc = first_of(gq);
      int rbb, kcc, pu; piece_of(pc, rbb, kcc, pu);
      const unsigned char* s = p.x + (size_t)(rbb * TM + pu * 8) * KD + kcc * KC;
      asm volatile("s_nop 4\n\tglobal_load_dwordx4 %0, %2, %3\n\tglobal_load_dwordx4 %1, %2, %4"
                   : "=&v"(dd[0]), "=&v"(dd[1]) : "v"(vsrc), "s"(s), "s"(s + 4 * KD) : "memory");
      if (gq < (unsigned)NHEAD) {
        #pragma unroll
        for (int u = 1; u < GSZ; ++u)
          asm volatile("s_nop 4\n\tglobal_load_dwordx4 %0, %2, %3\n\tglobal_load_dwordx4 %1, %2, %4"
                       : "=&v"(dd[2 * u]), "=&v"(dd[2 * u + 1]) : "v"(vsrc), "s"(s + (size_t)u * 8 * KD), "s"(s + (size_t)(u * 8 + 4) * KD) : "memory");
      } else {
        #pragma unroll
        for (int u = 1; u < GSZ; ++u) asm volatile("" : "=v"(dd[2 * u]), "=v"(dd[2 * u + 1]));
      }
    };
    const unsigned pw = (idx - 28) * 8 + xcd;
    if (pw >= (unsigned)NPA) return;
    unsigned cur = pw, nxt = pw + NPA;
    unsigned c;
    asm volatile("global_load_dword %0, %1, off sc1" : "=v"(c) : "v"(ctr) : "memory");
    load(cur, d[0]);
    wait_vm<0>();
    asm volatile("" : "+v"(c));
    const unsigned call = __builtin_amdgcn_readfirstlane(c) + 1;
    const unsigned base = (call - 1) * (unsigned)NGRP - 2 * NPA;
    const unsigned char* dbuf = peer + WS_XBUF + (size_t)(call & 1) * XBUF_BYTES + (size_t)rank * MS * KD;
    STAMP(1);
    unsigned char* pend = nullptr;
    auto st1 = [&](const unsigned char* dst, v4u v) {
      asm volatile("s_nop 4\n\tglobal_store_dwordx4 %0, %1, %2 sc0 sc1\n\ts_nop 1" :: "v"(lane * 16), "v"(v), "s"(dst) : "memory");
    };
    auto step = [&](v4u* dc, v4u* dn) {
      unsigned t = 0; const unsigned one = 1;
      const int pc = first_of(cur);
      int rbb, kcc, pu; piece_of(pc, rbb, kcc, pu);
      const unsigned char* dst = dbuf + (size_t)(rbb * NKC + kcc) * XT + pu * 2048;
      unsigned char* flag = peer + WS_FLAGS + ((size_t)((rank * NRB + rbb) * NKC + kcc) * PPU + pu) * 4;
      const bool head = cur < (unsigned)NHEAD;
      if (pend != nullptr) wait_vm<1>();
      #pragma unroll
      for (int u = 0; u < 2 * GSZ; ++u) asm volatile("" : "+v"(dc[u]));
      st1(dst, dc[0]);
      if (pend != nullptr) { wait_vm<1>(); if (lane == 0) st4_sys(pend, call); }
      st1(dst + 1024, dc[1]);
      if (lane == 0) asm volatile("global_atomic_add %0, %1, %2, off sc0" : "=v"(t) : "v"(tick), "v"(one) : "memory");
      if (nxt < (unsigned)NGRP) load(nxt, dn);
      else {
        #pragma unroll
        for (int u = 0; u < 2 * GSZ; ++u) asm volatile("" : "=v"(dn[u]));
      }
      if (head && nxt < (unsigned)NHEAD) {
        if (pend != nullptr) wait_vm<3 + 2 * GSZ>(); else wait_vm<2 + 2 * GSZ>();
        st1(dst + 2048, dc[2]);
        wait_vm<2 + 2 * GSZ>();
        st1(dst + 3072, dc[3]);
        #pragma unroll
        for (int u = 4; u < 2 * GSZ; ++u) { wait_vm<1>(); st1(dst + u * 1024, dc[u]); }
        wait_vm<2>();
        pend = flag;
      } else {
        wait_vm<0>();
        if (head) {
          #pragma unroll
          for (int u = 2; u < 2 * GSZ; u += 2) { st1(dst + u * 1024, dc[u]); st1(dst + (u + 1) * 1024, dc[u + 1]); wait_vm<0>(); }
        }
        if (lane == 0) st4_sys(flag, call);
        pend = nullptr;
      }
      asm volatile("" : "+v"(t));
      cur = nxt;
      nxt = __builtin_amdgcn_readfirstlane(t) - base;
    };
    while (cur < (unsigned)NGRP) {
      step(d[0], d[1]);
      if (cur >= (unsigned)NGRP) break;
      step(d[1], d[0]);
    }
    STAMP(2);
    if (lane == 0) asm volatile("global_store_dword %0, %1, off sc1" :: "v"(ctr), "v"(call) : "memory");
    return;
  }
  if (wid >= 6) return;

  const int nt = idx & 3;
  const int jj = idx >> 2;
  const int R = xcd * 7 + jj;
  const int rsrc = (rank + 1 + (R >> 3)) & 7;
  const int rrb = R & 7;
  const int npass = (jj == 0) ? 2 : 1;

  if (tid < 16) *(LAS unsigned*)(smem + LDS_CTL + tid * 4) = 0;
  asm volatile("s_waitcnt lgkmcnt(0)" ::: "memory");
  __builtin_amdgcn_s_barrier();

  const int ntot = npass * NKC;
  auto wait_free = [&](int c) {
    if (c >= STAGES) {
      for (;;) {
        unsigned f0 = lds_rd(ctl + (C_FREE + 0) * 4), f1 = lds_rd(ctl + (C_FREE + 1) * 4);
        unsigned f2 = lds_rd(ctl + (C_FREE + 2) * 4), f3 = lds_rd(ctl + (C_FREE + 3) * 4);
        unsigned m = min(min(f0, f1), min(f2, f3));
        if (m >= (unsigned)(c - STAGES + 1)) break;
        __builtin_amdgcn_s_sleep(1);
      }
    }
  };

  if (wid == 4) {
    unsigned* ctr = (unsigned*)(myws + WS_CTR_L) + bid;
    unsigned call;
    {
      unsigned c;
      asm volatile("global_load_dword %0, %1, off sc1\n\ts_waitcnt vmcnt(0)" : "=v"(c) : "v"(ctr) : "memory");
      call = __builtin_amdgcn_readfirstlane(c) + 1;
      if (lane == 0) asm volatile("global_store_dword %0, %1, off sc1" :: "v"(ctr), "v"(call) : "memory");
    }
    const unsigned char* fl = myws + WS_FLAGS + (size_t)(rsrc * NRB + rrb) * NKC * PPU * 4;
    unsigned* tmo = (unsigned*)(myws + WS_TMO);
    const int lr = lane >> 4, lc = lane & 15;
    bool dead = false;
    const unsigned long long t0 = __builtin_amdgcn_s_memrealtime();
    auto await = [&](int word, int pubw, unsigned pubv, bool nap) {
      bool first = true;
      for (;;) {
        unsigned f, tv;
        asm volatile("global_load_dword %0, %2, off sc0 sc1\n\tglobal_load_dword %1, %3, off sc1\n\ts_waitcnt vmcnt(0)"
                     : "=&v"(f), "=&v"(tv) : "v"(fl + word * 4), "v"(tmo) : "memory");
        if (first) { first = false; if (pubw >= 0) lds_wr(ctl + pubw * 4, pubv); }
        if (__all(f >= call) || dead) break;
        if (__builtin_amdgcn_readfirstlane(tv) == call || __builtin_amdgcn_s_memrealtime() - t0 > DEADLINE_TICKS) {
          if (lane == 0) asm volatile("global_store_dword %0, %1, off sc1" :: "v"(tmo), "v"(call) : "memory");
          lds_wr(ctl + C_BAD * 4, 1);
          dead = true;
          break;
        }
        if (nap) __builtin_amdgcn_s_sleep(2);
      }
    };
    for (int c = 0; c < ntot; ++c) {
      const int slot = c % STAGES;
      const bool own = (npass == 2 && c < NKC);
      const int kc = c & (NKC - 1);
      const unsigned char* xbase = own ? p.x + (size_t)xcd * TM * KD + kc * KC
                                       : myws + WS_XBUF + (size_t)(call & 1) * XBUF_BYTES + (size_t)((rsrc * NRB + rrb) * NKC + kc) * XT;
      const int rstride = own ? KD : KC;
      const unsigned dst = lds0 + LDS_X + slot * XT;
      wait_free(c);
      if (c == ntot - 1) {
        int issued = 0;
        bool first = true;
        while (issued < 4) {
          unsigned f, tv;
          asm volatile("global_load_dword %0, %2, off sc0 sc1\n\tglobal_load_dword %1, %3, off sc1\n\ts_waitcnt vmcnt(0)"
                       : "=&v"(f), "=&v"(tv) : "v"(fl + (kc * PPU + (lane & 7)) * 4), "v"(tmo) : "memory");
          if (first) { first = false; lds_wr(ctl + (C_FULLX + (c - 1) % STAGES) * 4, c); }
          else lds_wr(ctl + C_SUB * 4, issued);
          unsigned up = (unsigned)__ballot(f >= call) & 0xffu;
          if (up != 0xffu && !dead &&
              (__builtin_amdgcn_readfirstlane(tv) == call || __builtin_amdgcn_s_memrealtime() - t0 > DEADLINE_TICKS)) {
            if (lane == 0) asm volatile("global_store_dword %0, %1, off sc1" :: "v"(tmo), "v"(call) : "memory");
            lds_wr(ctl + C_BAD * 4, 1);
            dead = true;
          }
          if (dead) up = 0xffu;
          const int ready = __builtin_ctz(~up) >> 1;
          #pragma unroll
          for (int m = 0; m < 4; ++m) {
            if (m >= issued && m < ready) {
              #pragma unroll
              for (int i = 4 * m; i < 4 * m + 4; ++i) {
                const int row = 4 * i + lr;
                glds16_sc1(xbase + row * rstride + ((lc ^ (row & 15)) * 16), dst + i * 1024);
              }
            }
          }
          if (ready > issued) issued = ready;
        }
        STAMP(1);
        break;
      }
      if (!own) await(kc * PPU + (lane & (kc < NKC - FINEK ? PPU - GSZ : PPU - 1)), c > 0 ? C_FULLX + (c - 1) % STAGES : -1, c, true);
      if (c == ntot - NKC) STAMP(4);
      #pragma unroll
      for (int i = 0; i < 16; ++i) {
        const int row = 4 * i + lr;
        glds16_sc1(xbase + row * rstride + ((lc ^ (row & 15)) * 16), dst + i * 1024);
      }
      if (own && c > 0) { WAIT_VM(16); lds_wr(ctl + (C_FULLX + (c - 1) % STAGES) * 4, c); }
    }
    WAIT_VM(0);
    lds_wr(ctl + C_SUB * 4, 4);
    STAMP(2);
    return;
  }

  if (wid == 5) {
    const unsigned char* wbase = p.w + nt * TN;
    const int lr = lane >> 3, lc = lane & 7;
    for (int c = 0; c < ntot; ++c) {
      const int slot = c % STAGES;
      const int kc = c & (NKC - 1);
      wait_free(c);
      const unsigned dst = lds0 + LDS_W + slot * WT;
      #pragma unroll
      for (int i = 0; i < 32; ++i) {
        const int row = 8 * i + lr;
        const int h = ((row >> 1) & 3) | (((row >> 5) & 1) << 2);
        const unsigned char* g = wbase + (size_t)(kc * KC + row) * NL + ((lc ^ h) * 16);
        glds16(g, dst + i * 1024);
      }
      if (c > 0) { WAIT_VM(32); lds_wr(ctl + (C_FULLW + (c - 1) % STAGES) * 4, c); }
    }
    WAIT_VM(0);
    lds_wr(ctl + (C_FULLW + (ntot - 1) % STAGES) * 4, ntot);
    return;
  }

  const float scale = p.sx[0] * p.sw[0];
  const int g = lane >> 4, li = lane & 15;
  const int q = li >> 1, pp = li & 1;
  auto wfrag = [&](const LAS unsigned char* wsm, int ks, int n) -> v8i {
    v8i a;
    #pragma unroll
    for (int t = 0; t < 4; ++t) {
      const int k = 128 * ks + 32 * g + 8 * t + q;
      const int h = ((k >> 1) & 3) | (((k >> 5) & 1) << 2);
      v2i r = __builtin_amdgcn_ds_read_tr8_b64_v2i32((LAS v2i*)(wsm + k * 128 + (((2 * wid + n) ^ h) * 16) + 8 * pp));
      a[2 * t] = r[0]; a[2 * t + 1] = r[1];
    }
    return a;
  };
  auto xfrag = [&](const LAS unsigned char* xs, int ks, int m) -> v8i {
    const int row = 16 * m + li;
    v4i lo = *(const LAS v4i*)(xs + row * 256 + (((8 * ks + 2 * g) ^ li) * 16));
    v4i hi = *(const LAS v4i*)(xs + row * 256 + (((8 * ks + 2 * g + 1) ^ li) * 16));
    return v8i{lo[0], lo[1], lo[2], lo[3], hi[0], hi[1], hi[2], hi[3]};
  };
  auto put = [&](float* o, v4f v, bool bad) {
    v *= scale;
    v[0] = fmaxf(v[0], 0.f); v[1] = fmaxf(v[1], 0.f); v[2] = fmaxf(v[2], 0.f); v[3] = fmaxf(v[3], 0.f);
    if (bad) v = v4f{__builtin_nanf(""), __builtin_nanf(""), __builtin_nanf(""), __builtin_nanf("")};
    st16_sys(o, __builtin_bit_cast(v4u, v));
  };
  for (int pass = 0; pass < npass; ++pass) {
    const bool own = (npass == 2 && pass == 0);
    const int orow = own ? rank * MS + xcd * TM : rsrc * MS + rrb * TM;
    float* obase = p.out + (size_t)(orow + li) * NL + nt * TN + 32 * wid + 4 * g;
    v4f acc[4][2];
    #pragma unroll
    for (int m = 0; m < 4; ++m) { acc[m][0] = v4f{0, 0, 0, 0}; acc[m][1] = v4f{0, 0, 0, 0}; }
    const int nfull = own ? NKC : NKC - 1;
    for (int kc = 0; kc < nfull; ++kc) {
      const int c = pass * NKC + kc;
      const int slot = c % STAGES;
      for (;;) {
        unsigned fx = lds_rd(ctl + (C_FULLX + slot) * 4), fw = lds_rd(ctl + (C_FULLW + slot) * 4);
        if (min(fx, fw) >= (unsigned)(c + 1)) break;
        __builtin_amdgcn_s_sleep(1);
      }
      const LAS unsigned char* xs = (const LAS unsigned char*)smem + LDS_X + slot * XT;
      const LAS unsigned char* wsm = (const LAS unsigned char*)smem + LDS_W + slot * WT;
      #pragma unroll
      for (int ks = 0; ks < 2; ++ks) {
        v8i af[2], bf[4];
        #pragma unroll
        for (int n = 0; n < 2; ++n) af[n] = wfrag(wsm, ks, n);
        #pragma unroll
        for (int m = 0; m < 4; ++m) bf[m] = xfrag(xs, ks, m);
        if (ks == 1) {
          asm volatile("s_waitcnt lgkmcnt(0)" ::: "memory");
          __builtin_amdgcn_sched_barrier(0);
          if (c + STAGES < ntot) lds_wr(ctl + (C_FREE + wid) * 4, c + 1);
        }
        #pragma unroll
        for (int m = 0; m < 4; ++m)
          #pragma unroll
          for (int n = 0; n < 2; ++n)
            acc[m][n] = __builtin_amdgcn_mfma_scale_f32_16x16x128_f8f6f4(af[n], bf[m], acc[m][n], 1, 0, 0, 0, 0, 0);
      }
    }
    if (own) {
      #pragma unroll
      for (int m = 0; m < 4; ++m) { put(obase + (size_t)m * 16 * NL, acc[m][0], false); put(obase + (size_t)m * 16 * NL + 16, acc[m][1], false); }
    } else {
      const int slot = (ntot - 1) % STAGES;
      while (lds_rd(ctl + (C_FULLW + slot) * 4) < (unsigned)ntot) __builtin_amdgcn_s_sleep(1);
      const LAS unsigned char* xs = (const LAS unsigned char*)smem + LDS_X + slot * XT;
      const LAS unsigned char* wsm = (const LAS unsigned char*)smem + LDS_W + slot * WT;
      const v8i a00 = wfrag(wsm, 0, 0), a01 = wfrag(wsm, 0, 1), a10 = wfrag(wsm, 1, 0), a11 = wfrag(wsm, 1, 1);
      #pragma unroll
      for (int m = 0; m < 4; ++m) {
        while (lds_rd(ctl + C_SUB * 4) < (unsigned)(m + 1)) {}
        const bool bad = lds_rd(ctl + C_BAD * 4) != 0;
        const v8i b0 = xfrag(xs, 0, m), b1 = xfrag(xs, 1, m);
        acc[m][0] = __builtin_amdgcn_mfma_scale_f32_16x16x128_f8f6f4(a00, b0, acc[m][0], 1, 0, 0, 0, 0, 0);
        acc[m][1] = __builtin_amdgcn_mfma_scale_f32_16x16x128_f8f6f4(a01, b0, acc[m][1], 1, 0, 0, 0, 0, 0);
        acc[m][0] = __builtin_amdgcn_mfma_scale_f32_16x16x128_f8f6f4(a10, b1, acc[m][0], 1, 0, 0, 0, 0, 0);
        acc[m][1] = __builtin_amdgcn_mfma_scale_f32_16x16x128_f8f6f4(a11, b1, acc[m][1], 1, 0, 0, 0, 0, 0);
        if (m == 3) STAMP(1);
        put(obase + (size_t)m * 16 * NL, acc[m][0], bad); put(obase + (size_t)m * 16 * NL + 16, acc[m][1], bad);
      }
    }
    STAMP(2);
  }
}

}

extern "C" void kernel_launch(int rank, int nranks, ncclComm_t comm,
                              hipStream_t stream,
                              void* const* d_in, const int* in_sizes, int n_in,
                              void* d_out, int out_size,
                              const int* in_dtypes, int out_dtype,
                              void* const* d_out_all,
                              void* const* d_ws_all, size_t ws_bytes) {
  (void)comm; (void)in_sizes; (void)in_dtypes; (void)out_dtype; (void)d_out_all;
  if (nranks != NR || n_in != 4 || out_size != NR * MS * NL || ws_bytes < WS_NEED || (unsigned)rank >= (unsigned)NR) {
    fprintf(stderr, "kernel_launch: unexpected problem shape\n");
    return;
  }
  HIP_CHECK(hipFuncSetAttribute((const void*)ag_gemm_kernel, hipFuncAttributeMaxDynamicSharedMemorySize, LDS_BYTES));
  Params p;
  p.x = (const unsigned char*)d_in[0];
  p.w = (const unsigned char*)d_in[1];
  p.sx = (const float*)d_in[2];
  p.sw = (const float*)d_in[3];
  p.out = (float*)d_out;
  for (int i = 0; i < NR; ++i) p.ws[i] = (unsigned char*)d_ws_all[i];
  p.rank = rank;
  ag_gemm_kernel<<<NWG, 448, LDS_BYTES, stream>>>(p);
  HIP_CHECK(hipGetLastError());
}
```

```cpp
#include <hip/hip_runtime.h>
#include <rccl/rccl.h>
#include <cstdio>
#include <cstdint>

#define HIP_CHECK(expr) do { hipError_t _e = (expr); if (_e != hipSuccess) { \
    fprintf(stderr, "HIP error %s at %s:%d\n", hipGetErrorString(_e), __FILE__, __LINE__); \
    return; } } while (0)

namespace {

typedef int v2i __attribute__((ext_vector_type(2)));
typedef int v4i __attribute__((ext_vector_type(4)));
typedef int v8i __attribute__((ext_vector_type(8)));
typedef unsigned v4u __attribute__((ext_vector_type(4)));
typedef float v4f __attribute__((ext_vector_type(4)));
#define LAS __attribute__((address_space(3)))

constexpr int NR = 8;
constexpr int KD = 4096, NL = 512, MS = 512;
constexpr int TM = 64, TN = 128, KC = 256;
constexpr int NKC = KD / KC;
constexpr int NRB = MS / TM;
constexpr int NWG = 256;
constexpr int STAGES = 3;
constexpr int XT = TM * KC;
constexpr int WT = KC * TN;
constexpr int LDS_X = 0;
constexpr int LDS_W = STAGES * XT;
constexpr int LDS_CTL = STAGES * (XT + WT);
constexpr int LDS_BYTES = LDS_CTL + 256;
constexpr int C_FULLX = 0, C_FULLW = 4, C_FREE = 8  , C_BAD = 12, C_SUB = 13;

constexpr int NPWG = 32;
constexpr int NPA = 30;
constexpr int GSZ = 2;
constexpr int FINEK = 3;
constexpr int PPU = 8;
constexpr int NHEAD = (NKC - FINEK) * NRB * (PPU / GSZ);
constexpr int NGRP = NHEAD + FINEK * NRB * PPU;

constexpr size_t WS_CTR_L = 0;
constexpr size_t WS_CTR_P = 4096;
constexpr size_t WS_TMO = 24576;
constexpr size_t WS_TICK = 28672;
constexpr size_t WS_FLAGS = 65536;
constexpr size_t WS_XBUF = 1u << 20;
constexpr size_t XBUF_BYTES = (size_t)NR * MS * KD;
constexpr size_t WS_NEED = WS_XBUF + 2 * XBUF_BYTES;
constexpr unsigned long long DEADLINE_TICKS = 40000000ull;

struct Params {
  const unsigned char* x;
  const unsigned char* w;
  const float* sx;
  const float* sw;
  float* out;
  unsigned char* ws[NR];
  int rank;
};

__device__ __forceinline__ void glds16(const void* gsrc, unsigned lds_dst) {
  unsigned keep;
  asm volatile("s_mov_b32 %0, m0\n\ts_mov_b32 m0, %2\n\ts_nop 0\n\tglobal_load_lds_dwordx4 %1, off\n\ts_mov_b32 m0, %0"
               : "=&s"(keep) : "v"(gsrc), "s"(lds_dst) : "memory");
}
__device__ __forceinline__ void glds16_sc1(const void* gsrc, unsigned lds_dst) {
  unsigned keep;
  asm volatile("s_mov_b32 %0, m0\n\ts_mov_b32 m0, %2\n\ts_nop 0\n\tglobal_load_lds_dwordx4 %1, off sc1\n\ts_mov_b32 m0, %0"
               : "=&s"(keep) : "v"(gsrc), "s"(lds_dst) : "memory");
}
__device__ __forceinline__ unsigned lds_rd(unsigned addr) {
  unsigned v;
  asm volatile("ds_read_b32 %0, %1\n\ts_waitcnt lgkmcnt(0)" : "=v"(v) : "v"(addr) : "memory");
  return __builtin_amdgcn_readfirstlane(v);
}
__device__ __forceinline__ void lds_wr(unsigned addr, unsigned v) {
  asm volatile("ds_write_b32 %0, %1\n\ts_waitcnt lgkmcnt(0)" :: "v"(addr), "v"(v) : "memory");
}
__device__ __forceinline__ void st16_sys(void* p, v4u v) {
  asm volatile("global_store_dwordx4 %0, %1, off sc0 sc1\n\ts_nop 1" :: "v"(p), "v"(v) : "memory");
}
__device__ __forceinline__ void st4_sys(void* p, unsigned v) {
  asm volatile("global_store_dword %0, %1, off sc0 sc1" :: "v"(p), "v"(v) : "memory");
}
#define WAIT_VM(n) asm volatile("s_waitcnt vmcnt(" #n ")" ::: "memory")
template <int N> __device__ __forceinline__ void wait_vm() {
  asm volatile("s_waitcnt vmcnt(%0)" :: "i"(N > 63 ? 63 : N) : "memory");
}
__device__ __forceinline__ void piece_of(int pc, int& rb, int& kc, int& pu) {
  constexpr int HEAD = (NKC - 1) * NRB * PPU;
  if (pc < HEAD) { kc = pc / (NRB * PPU); rb = (pc / PPU) % NRB; pu = pc % PPU; }
  else { const int v = pc - HEAD; kc = NKC - 1; rb = (v >> 1) & 7; pu = (v >> 4) * 2 + (v & 1); }
}

__global__ void __launch_bounds__(448, 2) ag_gemm_kernel(Params p) {
  extern __shared__ __attribute__((aligned(16))) unsigned char smem[];
  const int tid = threadIdx.x;
  const int wid = __builtin_amdgcn_readfirstlane(tid >> 6);
  const int lane = tid & 63;
  const int bid = blockIdx.x;
  const int rank = p.rank;
  const unsigned lds0 = (unsigned)(uintptr_t)(LAS unsigned char*)smem;
  const unsigned ctl = lds0 + LDS_CTL;
  unsigned char* myws = p.ws[rank];

  const int xcd = bid & 7;
  const int idx = ((bid >> 3) + 28) & 31;

  if (idx >= 28) {
    const unsigned vsrc = (lane >> 4) * KD + (lane & 15) * 16;
    unsigned char* peer = p.ws[(rank + 1 + wid) & 7];
    unsigned* ctr = (unsigned*)(myws + WS_CTR_P) + bid * 16 + wid;
    unsigned* tick = (unsigned*)(myws + WS_TICK + wid * 128);
    v4u d[2][2 * GSZ];
    auto first_of = [&](unsigned gq) -> int { return gq < (unsigned)NHEAD ? gq * GSZ : NHEAD * GSZ + (gq - NHEAD); };
    auto load = [&](unsigned gq, v4u* dd) {
      const int pc = first_of(gq);
      int rbb, kcc, pu; piece_of(pc, rbb, kcc, pu);
      const unsigned char* s = p.x + (size_t)(rbb * TM + pu * 8) * KD + kcc * KC;
      asm volatile("s_nop 4\n\tglobal_load_dwordx4 %0, %2, %3\n\tglobal_load_dwordx4 %1, %2, %4"
                   : "=&v"(dd[0]), "=&v"(dd[1]) : "v"(vsrc), "s"(s), "s"(s + 4 * KD) : "memory");
      if (gq < (unsigned)NHEAD) {
        #pragma unroll
        for (int u = 1; u < GSZ; ++u)
          asm volatile("s_nop 4\n\tglobal_load_dwordx4 %0, %2, %3\n\tglobal_load_dwordx4 %1, %2, %4"
                       : "=&v"(dd[2 * u]), "=&v"(dd[2 * u + 1]) : "v"(vsrc), "s"(s + (size_t)u * 8 * KD), "s"(s + (size_t)(u * 8 + 4) * KD) : "memory");
      } else {
        #pragma unroll
        for (int u = 1; u < GSZ; ++u) asm volatile("" : "=v"(dd[2 * u]), "=v"(dd[2 * u + 1]));
      }
    };
    const unsigned pw = (idx - 28) * 8 + xcd;
    if (pw >= (unsigned)NPA) return;
    unsigned cur = pw, nxt = pw + NPA;
    unsigned c;
    asm volatile("global_load_dword %0, %1, off sc1" : "=v"(c) : "v"(ctr) : "memory");
    load(cur, d[0]);
    wait_vm<0>();
    asm volatile("" : "+v"(c));
    const unsigned call = __builtin_amdgcn_readfirstlane(c) + 1;
    const unsigned base = (call - 1) * (unsigned)NGRP - 2 * NPA;
    const unsigned char* dbuf = peer + WS_XBUF + (size_t)(call & 1) * XBUF_BYTES + (size_t)rank * MS * KD;
    unsigned char* pend = nullptr;
    auto st1 = [&](const unsigned char* dst, v4u v) {
      asm volatile("s_nop 4\n\tglobal_store_dwordx4 %0, %1, %2 sc0 sc1\n\ts_nop 1" :: "v"(lane * 16), "v"(v), "s"(dst) : "memory");
    };
    auto step = [&](v4u* dc, v4u* dn) {
      unsigned t = 0; const unsigned one = 1;
      const int pc = first_of(cur);
      int rbb, kcc, pu; piece_of(pc, rbb, kcc, pu);
      const unsigned char* dst = dbuf + (size_t)(rbb * NKC + kcc) * XT + pu * 2048;
      unsigned char* flag = peer + WS_FLAGS + ((size_t)((rank * NRB + rbb) * NKC + kcc) * PPU + pu) * 4;
      const bool head = cur < (unsigned)NHEAD;
      if (pend != nullptr) wait_vm<1>();
      #pragma unroll
      for (int u = 0; u < 2 * GSZ; ++u) asm volatile("" : "+v"(dc[u]));
      st1(dst, dc[0]);
      if (pend != nullptr) { wait_vm<1>(); if (lane == 0) st4_sys(pend, call); }
      st1(dst + 1024, dc[1]);
      if (lane == 0) asm volatile("global_atomic_add %0, %1, %2, off sc0" : "=v"(t) : "v"(tick), "v"(one) : "memory");
      if (nxt < (unsigned)NGRP) load(nxt, dn);
      else {
        #pragma unroll
        for (int u = 0; u < 2 * GSZ; ++u) asm volatile("" : "=v"(dn[u]));
      }
      if (head && nxt < (unsigned)NHEAD) {
        if (pend != nullptr) wait_vm<3 + 2 * GSZ>(); else wait_vm<2 + 2 * GSZ>();
        st1(dst + 2048, dc[2]);
        wait_vm<2 + 2 * GSZ>();
        st1(dst + 3072, dc[3]);
        #pragma unroll
        for (int u = 4; u < 2 * GSZ; ++u) { wait_vm<1>(); st1(dst + u * 1024, dc[u]); }
        wait_vm<2>();
        pend = flag;
      } else {
        wait_vm<0>();
        if (head) {
          #pragma unroll
          for (int u = 2; u < 2 * GSZ; u += 2) { st1(dst + u * 1024, dc[u]); st1(dst + (u + 1) * 1024, dc[u + 1]); wait_vm<0>(); }
        }
        if (lane == 0) st4_sys(flag, call);
        pend = nullptr;
      }
      asm volatile("" : "+v"(t));
      cur = nxt;
      nxt = __builtin_amdgcn_readfirstlane(t) - base;
    };
    while (cur < (unsigned)NGRP) {
      step(d[0], d[1]);
      if (cur >= (unsigned)NGRP) break;
      step(d[1], d[0]);
    }
    if (lane == 0) asm volatile("global_store_dword %0, %1, off sc1" :: "v"(ctr), "v"(call) : "memory");
    return;
  }
  if (wid >= 6) return;

  const int nt = idx & 3;
  const int jj = idx >> 2;
  const int R = xcd * 7 + jj;
  const int rsrc = (rank + 1 + (R >> 3)) & 7;
  const int rrb = R & 7;
  const int npass = (jj == 0) ? 2 : 1;

  if (tid < 16) *(LAS unsigned*)(smem + LDS_CTL + tid * 4) = 0;
  asm volatile("s_waitcnt lgkmcnt(0)" ::: "memory");
  __builtin_amdgcn_s_barrier();

  const int ntot = npass * NKC;
  auto wait_free = [&](int c) {
    if (c >= STAGES) {
      for (;;) {
        unsigned f0 = lds_rd(ctl + (C_FREE + 0) * 4), f1 = lds_rd(ctl + (C_FREE + 1) * 4);
        unsigned f2 = lds_rd(ctl + (C_FREE + 2) * 4), f3 = lds_rd(ctl + (C_FREE + 3) * 4);
        unsigned m = min(min(f0, f1), min(f2, f3));
        if (m >= (unsigned)(c - STAGES + 1)) break;
        __builtin_amdgcn_s_sleep(1);
      }
    }
  };

  if (wid == 4) {
    unsigned* ctr = (unsigned*)(myws + WS_CTR_L) + bid;
    unsigned call;
    {
      unsigned c;
      asm volatile("global_load_dword %0, %1, off sc1\n\ts_waitcnt vmcnt(0)" : "=v"(c) : "v"(ctr) : "memory");
      call = __builtin_amdgcn_readfirstlane(c) + 1;
      if (lane == 0) asm volatile("global_store_dword %0, %1, off sc1" :: "v"(ctr), "v"(call) : "memory");
    }
    const unsigned char* fl = myws + WS_FLAGS + (size_t)(rsrc * NRB + rrb) * NKC * PPU * 4;
    unsigned* tmo = (unsigned*)(myws + WS_TMO);
    const int lr = lane >> 4, lc = lane & 15;
    bool dead = false;
    const unsigned long long t0 = __builtin_amdgcn_s_memrealtime();
    auto await = [&](int word, int pubw, unsigned pubv, bool nap) {
      bool first = true;
      for (;;) {
        unsigned f, tv;
        asm volatile("global_load_dword %0, %2, off sc0 sc1\n\tglobal_load_dword %1, %3, off sc1\n\ts_waitcnt vmcnt(0)"
                     : "=&v"(f), "=&v"(tv) : "v"(fl + word * 4), "v"(tmo) : "memory");
        if (first) { first = false; if (pubw >= 0) lds_wr(ctl + pubw * 4, pubv); }
        if (__all(f >= call) || dead) break;
        if (__builtin_amdgcn_readfirstlane(tv) == call || __builtin_amdgcn_s_memrealtime() - t0 > DEADLINE_TICKS) {
          if (lane == 0) asm volatile("global_store_dword %0, %1, off sc1" :: "v"(tmo), "v"(call) : "memory");
          lds_wr(ctl + C_BAD * 4, 1);
          dead = true;
          break;
        }
        if (nap) __builtin_amdgcn_s_sleep(2);
      }
    };
    for (int c = 0; c < ntot; ++c) {
      const int slot = c % STAGES;
      const bool own = (npass == 2 && c < NKC);
      const int kc = c & (NKC - 1);
      const unsigned char* xbase = own ? p.x + (size_t)xcd * TM * KD + kc * KC
                                       : myws + WS_XBUF + (size_t)(call & 1) * XBUF_BYTES + (size_t)((rsrc * NRB + rrb) * NKC + kc) * XT;
      const int rstride = own ? KD : KC;
      const unsigned dst = lds0 + LDS_X + slot * XT;
      wait_free(c);
      if (c == ntot - 1) {
        int issued = 0;
        bool first = true;
        while (issued < 4) {
          unsigned f, tv;
          asm volatile("global_load_dword %0, %2, off sc0 sc1\n\tglobal_load_dword %1, %3, off sc1\n\ts_waitcnt vmcnt(0)"
                       : "=&v"(f), "=&v"(tv) : "v"(fl + (kc * PPU + (lane & 7)) * 4), "v"(tmo) : "memory");
          if (first) { first = false; lds_wr(ctl + (C_FULLX + (c - 1) % STAGES) * 4, c); }
          else lds_wr(ctl + C_SUB * 4, issued);
          unsigned up = (unsigned)__ballot(f >= call) & 0xffu;
          if (up != 0xffu && !dead &&
              (__builtin_amdgcn_readfirstlane(tv) == call || __builtin_amdgcn_s_memrealtime() - t0 > DEADLINE_TICKS)) {
            if (lane == 0) asm volatile("global_store_dword %0, %1, off sc1" :: "v"(tmo), "v"(call) : "memory");
            lds_wr(ctl + C_BAD * 4, 1);
            dead = true;
          }
          if (dead) up = 0xffu;
          const int ready = __builtin_ctz(~up) >> 1;
          #pragma unroll
          for (int m = 0; m < 4; ++m) {
            if (m >= issued && m < ready) {
              #pragma unroll
              for (int i = 4 * m; i < 4 * m + 4; ++i) {
                const int row = 4 * i + lr;
                glds16_sc1(xbase + row * rstride + ((lc ^ (row & 15)) * 16), dst + i * 1024);
              }
            }
          }
          if (ready > issued) issued = ready;
        }
        break;
      }
      if (!own) await(kc * PPU + (lane & (kc < NKC - FINEK ? PPU - GSZ : PPU - 1)), c > 0 ? C_FULLX + (c - 1) % STAGES : -1, c, true);
      #pragma unroll
      for (int i = 0; i < 16; ++i) {
        const int row = 4 * i + lr;
        glds16_sc1(xbase + row * rstride + ((lc ^ (row & 15)) * 16), dst + i * 1024);
      }
      if (own && c > 0) { WAIT_VM(16); lds_wr(ctl + (C_FULLX + (c - 1) % STAGES) * 4, c); }
    }
    WAIT_VM(0);
    lds_wr(ctl + C_SUB * 4, 4);
    return;
  }

  if (wid == 5) {
    const unsigned char* wbase = p.w + nt * TN;
    const int lr = lane >> 3, lc = lane & 7;
    for (int c = 0; c < ntot; ++c) {
      const int slot = c % STAGES;
      const int kc = c & (NKC - 1);
      wait_free(c);
      const unsigned dst = lds0 + LDS_W + slot * WT;
      #pragma unroll
      for (int i = 0; i < 32; ++i) {
        const int row = 8 * i + lr;
        const int h = ((row >> 1) & 3) | (((row >> 5) & 1) << 2);
        const unsigned char* g = wbase + (size_t)(kc * KC + row) * NL + ((lc ^ h) * 16);
        glds16(g, dst + i * 1024);
      }
      if (c > 0) { WAIT_VM(32); lds_wr(ctl + (C_FULLW + (c - 1) % STAGES) * 4, c); }
    }
    WAIT_VM(0);
    lds_wr(ctl + (C_FULLW + (ntot - 1) % STAGES) * 4, ntot);
    return;
  }

  const float scale = p.sx[0] * p.sw[0];
  const int g = lane >> 4, li = lane & 15;
  const int q = li >> 1, pp = li & 1;
  auto wfrag = [&](const LAS unsigned char* wsm, int ks, int n) -> v8i {
    v8i a;
    #pragma unroll
    for (int t = 0; t < 4; ++t) {
      const int k = 128 * ks + 32 * g + 8 * t + q;
      const int h = ((k >> 1) & 3) | (((k >> 5) & 1) << 2);
      v2i r = __builtin_amdgcn_ds_read_tr8_b64_v2i32((LAS v2i*)(wsm + k * 128 + (((2 * wid + n) ^ h) * 16) + 8 * pp));
      a[2 * t] = r[0]; a[2 * t + 1] = r[1];
    }
    return a;
  };
  auto xfrag = [&](const LAS unsigned char* xs, int ks, int m) -> v8i {
    const int row = 16 * m + li;
    v4i lo = *(const LAS v4i*)(xs + row * 256 + (((8 * ks + 2 * g) ^ li) * 16));
    v4i hi = *(const LAS v4i*)(xs + row * 256 + (((8 * ks + 2 * g + 1) ^ li) * 16));
    return v8i{lo[0], lo[1], lo[2], lo[3], hi[0], hi[1], hi[2], hi[3]};
  };
  auto put = [&](float* o, v4f v, bool bad) {
    v *= scale;
    v[0] = fmaxf(v[0], 0.f); v[1] = fmaxf(v[1], 0.f); v[2] = fmaxf(v[2], 0.f); v[3] = fmaxf(v[3], 0.f);
    if (bad) v = v4f{__builtin_nanf(""), __builtin_nanf(""), __builtin_nanf(""), __builtin_nanf("")};
    st16_sys(o, __builtin_bit_cast(v4u, v));
  };
  for (int pass = 0; pass < npass; ++pass) {
    const bool own = (npass == 2 && pass == 0);
    const int orow = own ? rank * MS + xcd * TM : rsrc * MS + rrb * TM;
    float* obase = p.out + (size_t)(orow + li) * NL + nt * TN + 32 * wid + 4 * g;
    v4f acc[4][2];
    #pragma unroll
    for (int m = 0; m < 4; ++m) { acc[m][0] = v4f{0, 0, 0, 0}; acc[m][1] = v4f{0, 0, 0, 0}; }
    const int nfull = own ? NKC : NKC - 1;
    for (int kc = 0; kc < nfull; ++kc) {
      const int c = pass * NKC + kc;
      const int slot = c % STAGES;
      for (;;) {
        unsigned fx = lds_rd(ctl + (C_FULLX + slot) * 4), fw = lds_rd(ctl + (C_FULLW + slot) * 4);
        if (min(fx, fw) >= (unsigned)(c + 1)) break;
        __builtin_amdgcn_s_sleep(1);
      }
      const LAS unsigned char* xs = (const LAS unsigned char*)smem + LDS_X + slot * XT;
      const LAS unsigned char* wsm = (const LAS unsigned char*)smem + LDS_W + slot * WT;
      #pragma unroll
      for (int ks = 0; ks < 2; ++ks) {
        v8i af[2], bf[4];
        #pragma unroll
        for (int n = 0; n < 2; ++n) af[n] = wfrag(wsm, ks, n);
        #pragma unroll
        for (int m = 0; m < 4; ++m) bf[m] = xfrag(xs, ks, m);
        if (ks == 1) {
          asm volatile("s_waitcnt lgkmcnt(0)" ::: "memory");
          __builtin_amdgcn_sched_barrier(0);
          if (c + STAGES < ntot) lds_wr(ctl + (C_FREE + wid) * 4, c + 1);
        }
        #pragma unroll
        for (int m = 0; m < 4; ++m)
          #pragma unroll
          for (int n = 0; n < 2; ++n)
            acc[m][n] = __builtin_amdgcn_mfma_scale_f32_16x16x128_f8f6f4(af[n], bf[m], acc[m][n], 1, 0, 0, 0, 0, 0);
      }
    }
    if (own) {
      #pragma unroll
      for (int m = 0; m < 4; ++m) { put(obase + (size_t)m * 16 * NL, acc[m][0], false); put(obase + (size_t)m * 16 * NL + 16, acc[m][1], false); }
    } else {
      const int slot = (ntot - 1) % STAGES;
      while (lds_rd(ctl + (C_FULLW + slot) * 4) < (unsigned)ntot) __builtin_amdgcn_s_sleep(1);
      const LAS unsigned char* xs = (const LAS unsigned char*)smem + LDS_X + slot * XT;
      const LAS unsigned char* wsm = (const LAS unsigned char*)smem + LDS_W + slot * WT;
      const v8i a00 = wfrag(wsm, 0, 0), a01 = wfrag(wsm, 0, 1), a10 = wfrag(wsm, 1, 0), a11 = wfrag(wsm, 1, 1);
      #pragma unroll
      for (int m = 0; m < 4; ++m) {
        while (lds_rd(ctl + C_SUB * 4) < (unsigned)(m + 1)) {}
        const bool bad = lds_rd(ctl + C_BAD * 4) != 0;
        const v8i b0 = xfrag(xs, 0, m), b1 = xfrag(xs, 1, m);
        acc[m][0] = __builtin_amdgcn_mfma_scale_f32_16x16x128_f8f6f4(a00, b0, acc[m][0], 1, 0, 0, 0, 0, 0);
        acc[m][1] = __builtin_amdgcn_mfma_scale_f32_16x16x128_f8f6f4(a01, b0, acc[m][1], 1, 0, 0, 0, 0, 0);
        acc[m][0] = __builtin_amdgcn_mfma_scale_f32_16x16x128_f8f6f4(a10, b1, acc[m][0], 1, 0, 0, 0, 0, 0);
        acc[m][1] = __builtin_amdgcn_mfma_scale_f32_16x16x128_f8f6f4(a11, b1, acc[m][1], 1, 0, 0, 0, 0, 0);
        put(obase + (size_t)m * 16 * NL, acc[m][0], bad); put(obase + (size_t)m * 16 * NL + 16, acc[m][1], bad);
      }
    }
  }
}

}

extern "C" void kernel_launch(int rank, int nranks, ncclComm_t comm,
                              hipStream_t stream,
                              void* const* d_in, const int* in_sizes, int n_in,
                              void* d_out, int out_size,
                              const int* in_dtypes, int out_dtype,
                              void* const* d_out_all,
                              void* const* d_ws_all, size_t ws_bytes) {
  (void)comm; (void)in_sizes; (void)in_dtypes; (void)out_dtype; (void)d_out_all;
  if (nranks != NR || n_in != 4 || out_size != NR * MS * NL || ws_bytes < WS_NEED || (unsigned)rank >= (unsigned)NR) {
    fprintf(stderr, "kernel_launch: unexpected problem shape\n");
    return;
  }
  HIP_CHECK(hipFuncSetAttribute((const void*)ag_gemm_kernel, hipFuncAttributeMaxDynamicSharedMemorySize, LDS_BYTES));
  Params p;
  p.x = (const unsigned char*)d_in[0];
  p.w = (const unsigned char*)d_in[1];
  p.sx = (const float*)d_in[2];
  p.sw = (const float*)d_in[3];
  p.out = (float*)d_out;
  for (int i = 0; i < NR; ++i) p.ws[i] = (unsigned char*)d_ws_all[i];
  p.rank = rank;
  ag_gemm_kernel<<<NWG, 448, LDS_BYTES, stream>>>(p);
  HIP_CHECK(hipGetLastError());
}
```
